```python
import numpy as np

kernel_name = "gpu_a2a_gemm_m4096_k4096_n32768_f32_relu"


ABSMAX_RTOL = 0.001

COMPUTE_DTYPE = "float32"
EPILOGUE = "relu"
IN_QUANT = None
OUT_QUANT = None
IN_DTYPES = {"x": "float32", "w_mat": "float32"}
WORLD_SIZE = 8

def setup_inputs():

    x = np.random.randn(4096, 4096).astype(np.float32)
    w_mat = (np.random.randn(4096, 32768) / np.sqrt(4096)).astype(np.float32)
    return {"x": x, "w_mat": w_mat}

def _epilogue(y):
    return np.maximum(y, 0.0).astype(np.float32)

def reference(x, w_mat):
    y = (x @ w_mat).astype(np.float32)
    return _epilogue(y)
```

```cpp
#include <hip/hip_runtime.h>
#include <rccl/rccl.h>
#include <cstdio>

#define HIP_CHECK(expr) do { hipError_t _e = (expr); if (_e != hipSuccess) { \
    fprintf(stderr, "HIP error %s at %s:%d\n", hipGetErrorString(_e), __FILE__, __LINE__); \
    return; } } while (0)

#define GEMM_ASM \
"s_memrealtime s[70:71]\n\t" \
"s_mov_b32 s36, %[wlo]\n\t" \
"s_mov_b32 s37, %[whi]\n\t" \
"s_mov_b32 s38, 0x20000000\n\t" \
"s_mov_b32 s39, 0x00020000\n\t" \
"s_mov_b32 s40, %[mlo]\n\t" \
"s_mov_b32 s41, %[mhi]\n\t" \
"s_mov_b32 s42, 0x400000\n\t" \
"s_mov_b32 s43, 0x00020000\n\t" \
"s_mov_b32 s44, %[olo]\n\t" \
"s_mov_b32 s45, %[ohi]\n\t" \
"s_mov_b32 s46, 0x4000000\n\t" \
"s_mov_b32 s47, 0x00020000\n\t" \
"s_mov_b32 s48, %[mlo]\n\t" \
"s_mov_b32 s49, %[mhi]\n\t" \
"s_mov_b32 s50, 0x10000\n\t" \
"s_mov_b32 s51, 0x00020000\n\t" \
"s_mov_b32 s52, %[dlo]\n\t" \
"s_mov_b32 s53, %[dhi]\n\t" \
"s_mov_b32 s54, 0x810000\n\t" \
"s_mov_b32 s55, 0x00020000\n\t" \
"s_mov_b32 s72, %[xilo]\n\t" \
"s_mov_b32 s73, %[xihi]\n\t" \
"s_mov_b32 s74, 0x800000\n\t" \
"s_mov_b32 s75, 0x00020000\n\t" \
"s_lshl_b32 s56, %[rank], 26\n\t" \
"s_lshl_b32 s61, %[rank], 19\n\t" \
"s_mov_b32 s62, 32\n\t" \
"s_mov_b32 s64, 2\n\t" \
"s_mov_b32 s88, 4\n\t" \
"s_mov_b32 s69, 0\n\t" \
"s_mov_b32 s80, %[mlo]\n\t" \
"s_mov_b32 s81, %[mhi]\n\t" \
"s_mov_b32 s83, 0x00020000\n\t" \
"s_cmp_eq_u32 %[wave], 0\n\t" \
"s_cselect_b32 s82, 4096, 0\n\t" \
"v_add_u32 v248, 0x18000, %[ldsrd]\n\t" \
"v_mov_b32 v249, %[ldswr]\n\t" \
"buffer_load_dwordx2 v[208:209], off, s[48:51], %[ctroff] sc1\n\t" \
"s_getpc_b64 s[76:77]\n\t" \
"CODE0_%=:\n\t" \
"s_mov_b32 s78, KLAST_%=-CODE0_%=\n\t" \
"s_mov_b32 s79, 0x00020000\n\t" \
"v_subrev_u32 v254, 131072, %[ldsstg]\n\t" \
"v_lshlrev_b32 v254, 2, v254\n\t" \
"buffer_load_dword v244, v254, s[76:79], 0 offen\n\t" \
"s_add_u32 s63, %[dstoff], 65536\n\t" \
"s_add_u32 s65, %[dstoff], 4259840\n\t" \
"buffer_load_dword v245, off, s[52:55], s63 sc0 sc1\n\t" \
"buffer_load_dword v246, off, s[52:55], s65 sc0 sc1\n\t" \
"s_mov_b32 s79, %[avinit]\n\t" \
"v_mov_b32 v204, 0\n\t" \
"v_mov_b32 v205, 0\n\t" \
"v_mov_b32 v206, 0\n\t" \
"v_mov_b32 v207, 0\n\t" \
"s_mov_b32 s89, 1720\n\t" \
"s_mov_b32 s90, 340\n\t" \
"s_mov_b32 s91, 8\n\t" \
"s_mov_b32 s94, 0xf0000000\n\t" \
"buffer_load_dwordx4 v[12:15], %[xioff], s[72:75], 0 offen offset:0 \n\t" \
"buffer_load_dwordx4 v[16:19], %[xioff], s[72:75], 0 offen offset:64 \n\t" \
"buffer_load_dwordx4 v[20:23], %[xioff], s[72:75], 0 offen offset:16 \n\t" \
"buffer_load_dwordx4 v[24:27], %[xioff], s[72:75], 0 offen offset:80 \n\t" \
"buffer_load_dwordx4 v[28:31], %[xioff], s[72:75], 0 offen offset:32 \n\t" \
"buffer_load_dwordx4 v[32:35], %[xioff], s[72:75], 0 offen offset:96 \n\t" \
"buffer_load_dwordx4 v[36:39], %[xioff], s[72:75], 0 offen offset:48 \n\t" \
"buffer_load_dwordx4 v[40:43], %[xioff], s[72:75], 0 offen offset:112 \n\t" \
"s_cmp_lg_u32 %[slot], 0\n\t" \
"s_cbranch_scc1 STAGE_%=\n\t" \
"v_mfma_f32_32x32x16_f16 a[0:15], v[204:207], v[204:207], 0\n\t" \
"v_mfma_f32_32x32x16_f16 a[16:31], v[204:207], v[204:207], 0\n\t" \
"v_mfma_f32_32x32x16_f16 a[32:47], v[204:207], v[204:207], 0\n\t" \
"v_mfma_f32_32x32x16_f16 a[48:63], v[204:207], v[204:207], 0\n\t" \
"v_mfma_f32_32x32x16_f16 a[64:79], v[204:207], v[204:207], 0\n\t" \
"v_mfma_f32_32x32x16_f16 a[80:95], v[204:207], v[204:207], 0\n\t" \
"v_mfma_f32_32x32x16_f16 a[96:111], v[204:207], v[204:207], 0\n\t" \
"v_mfma_f32_32x32x16_f16 a[112:127], v[204:207], v[204:207], 0\n\t" \
"v_mfma_f32_32x32x16_f16 a[128:143], v[204:207], v[204:207], 0\n\t" \
"v_mfma_f32_32x32x16_f16 a[144:159], v[204:207], v[204:207], 0\n\t" \
"v_mfma_f32_32x32x16_f16 a[160:175], v[204:207], v[204:207], 0\n\t" \
"v_mfma_f32_32x32x16_f16 a[176:191], v[204:207], v[204:207], 0\n\t" \
"v_mfma_f32_32x32x16_f16 a[192:207], v[204:207], v[204:207], 0\n\t" \
"v_mfma_f32_32x32x16_f16 a[208:223], v[204:207], v[204:207], 0\n\t" \
"v_mfma_f32_32x32x16_f16 a[224:239], v[204:207], v[204:207], 0\n\t" \
"v_mfma_f32_32x32x16_f16 a[240:255], v[204:207], v[204:207], 0\n\t" \
"s_waitcnt vmcnt(0) lgkmcnt(0)\n\t" \
"v_readfirstlane_b32 s68, v208\n\t" \
"v_readfirstlane_b32 s66, v209\n\t" \
"v_cvt_pk_f16_f32 v44, v12, v13\n\t" \
"v_cvt_pk_f16_f32 v45, v14, v15\n\t" \
"v_cvt_pk_f16_f32 v46, v16, v17\n\t" \
"v_cvt_pk_f16_f32 v47, v18, v19\n\t" \
"v_cvt_pk_f16_f32 v48, v20, v21\n\t" \
"v_cvt_pk_f16_f32 v49, v22, v23\n\t" \
"v_cvt_pk_f16_f32 v50, v24, v25\n\t" \
"v_cvt_pk_f16_f32 v51, v26, v27\n\t" \
"v_cvt_pk_f16_f32 v52, v28, v29\n\t" \
"v_cvt_pk_f16_f32 v53, v30, v31\n\t" \
"v_cvt_pk_f16_f32 v54, v32, v33\n\t" \
"v_cvt_pk_f16_f32 v55, v34, v35\n\t" \
"v_cvt_pk_f16_f32 v56, v36, v37\n\t" \
"v_cvt_pk_f16_f32 v57, v38, v39\n\t" \
"v_cvt_pk_f16_f32 v58, v40, v41\n\t" \
"v_cvt_pk_f16_f32 v59, v42, v43\n\t" \
"s_sub_u32 s66, s70, s66\n\t" \
"s_cmp_lt_u32 s66, 6000\n\t" \
"s_cselect_b32 s89, 0xf0000000, s89\n\t" \
"s_cselect_b32 s91, 0, s91\n\t" \
"s_add_u32 s68, s68, 1\n\t" \
"s_and_b32 s63, s68, 1\n\t" \
"s_lshl_b32 s63, s63, 22\n\t" \
"s_add_u32 s63, s63, 65536\n\t" \
"s_add_u32 s78, %[dstoff], s63\n\t" \
"s_add_u32 s40, s40, s63\n\t" \
"s_addc_u32 s41, s41, 0\n\t" \
"s_add_u32 s57, s78, 0\n\t" \
"buffer_store_dwordx4 v[44:47], %[pshoff], s[52:55], s57 offen sc0 sc1\n\t" \
"s_add_u32 s57, s78, 1024\n\t" \
"buffer_store_dwordx4 v[48:51], %[pshoff], s[52:55], s57 offen sc0 sc1\n\t" \
"s_add_u32 s57, s78, 16384\n\t" \
"buffer_store_dwordx4 v[52:55], %[pshoff], s[52:55], s57 offen sc0 sc1\n\t" \
"s_add_u32 s57, s78, 17408\n\t" \
"buffer_store_dwordx4 v[56:59], %[pshoff], s[52:55], s57 offen sc0 sc1\n\t" \
"v_mov_b32 v251, 1\n\t" \
"s_cmp_eq_u32 %[dst], %[rank]\n\t" \
"s_cbranch_scc1 OWNP_%=\n\t" \
"s_add_u32 s57, s56, 0x0\n\t" \
"buffer_load_dwordx4 v[140:143], %[woff], s[36:39], s57 offen nt\n\t" \
"s_add_u32 s58, s56, 0x20000\n\t" \
"buffer_load_dwordx4 v[144:147], %[woff], s[36:39], s58 offen nt\n\t" \
"s_add_u32 s57, s56, 0x40000\n\t" \
"buffer_load_dwordx4 v[148:151], %[woff], s[36:39], s57 offen nt\n\t" \
"s_add_u32 s58, s56, 0x60000\n\t" \
"buffer_load_dwordx4 v[152:155], %[woff], s[36:39], s58 offen nt\n\t" \
"s_add_u32 s57, s56, 0x200000\n\t" \
"buffer_load_dwordx4 v[156:159], %[woff], s[36:39], s57 offen nt\n\t" \
"s_add_u32 s58, s56, 0x220000\n\t" \
"buffer_load_dwordx4 v[160:163], %[woff], s[36:39], s58 offen nt\n\t" \
"s_add_u32 s57, s56, 0x240000\n\t" \
"buffer_load_dwordx4 v[164:167], %[woff], s[36:39], s57 offen nt\n\t" \
"s_add_u32 s58, s56, 0x260000\n\t" \
"buffer_load_dwordx4 v[168:171], %[woff], s[36:39], s58 offen nt\n\t" \
"s_add_u32 s57, s56, 0x800000\n\t" \
"buffer_load_dwordx4 v[172:175], %[woff], s[36:39], s57 offen nt\n\t" \
"s_add_u32 s58, s56, 0x820000\n\t" \
"buffer_load_dwordx4 v[176:179], %[woff], s[36:39], s58 offen nt\n\t" \
"s_add_u32 s57, s56, 0x840000\n\t" \
"buffer_load_dwordx4 v[180:183], %[woff], s[36:39], s57 offen nt\n\t" \
"s_add_u32 s58, s56, 0x860000\n\t" \
"buffer_load_dwordx4 v[184:187], %[woff], s[36:39], s58 offen nt\n\t" \
"s_add_u32 s57, s56, 0xa00000\n\t" \
"buffer_load_dwordx4 v[188:191], %[woff], s[36:39], s57 offen nt\n\t" \
"s_add_u32 s58, s56, 0xa20000\n\t" \
"buffer_load_dwordx4 v[192:195], %[woff], s[36:39], s58 offen nt\n\t" \
"s_add_u32 s57, s56, 0xa40000\n\t" \
"buffer_load_dwordx4 v[196:199], %[woff], s[36:39], s57 offen nt\n\t" \
"s_add_u32 s58, s56, 0xa60000\n\t" \
"buffer_load_dwordx4 v[200:203], %[woff], s[36:39], s58 offen nt\n\t" \
"s_waitcnt vmcnt(16)\n\t" \
"s_barrier\n\t" \
"s_cmp_lg_u32 %[wave], 0\n\t" \
"s_cbranch_scc1 PDONE_%=\n\t" \
"s_mov_b64 exec, 1\n\t" \
"buffer_atomic_add v251, off, s[52:55], %[flgoff] sc1\n\t" \
"s_mov_b64 exec, -1\n\t" \
"s_branch PDONE_%=\n\t" \
"OWNP_%=:\n\t" \
"s_waitcnt vmcnt(0)\n\t" \
"s_barrier\n\t" \
"s_cmp_lg_u32 %[wave], 0\n\t" \
"s_cbranch_scc1 OWNQ_%=\n\t" \
"s_mov_b64 exec, 1\n\t" \
"buffer_atomic_add v251, off, s[52:55], %[flgoff] sc1\n\t" \
"s_mov_b64 exec, -1\n\t" \
"OWNQ_%=:\n\t" \
"s_barrier\n\t" \
"s_add_u32 s57, s56, 0x0\n\t" \
"buffer_load_dwordx4 v[140:143], %[woff], s[36:39], s57 offen nt\n\t" \
"s_add_u32 s58, s56, 0x20000\n\t" \
"buffer_load_dwordx4 v[144:147], %[woff], s[36:39], s58 offen nt\n\t" \
"s_add_u32 s57, s56, 0x40000\n\t" \
"buffer_load_dwordx4 v[148:151], %[woff], s[36:39], s57 offen nt\n\t" \
"s_add_u32 s58, s56, 0x60000\n\t" \
"buffer_load_dwordx4 v[152:155], %[woff], s[36:39], s58 offen nt\n\t" \
"s_add_u32 s57, s56, 0x200000\n\t" \
"buffer_load_dwordx4 v[156:159], %[woff], s[36:39], s57 offen nt\n\t" \
"s_add_u32 s58, s56, 0x220000\n\t" \
"buffer_load_dwordx4 v[160:163], %[woff], s[36:39], s58 offen nt\n\t" \
"s_add_u32 s57, s56, 0x240000\n\t" \
"buffer_load_dwordx4 v[164:167], %[woff], s[36:39], s57 offen nt\n\t" \
"s_add_u32 s58, s56, 0x260000\n\t" \
"buffer_load_dwordx4 v[168:171], %[woff], s[36:39], s58 offen nt\n\t" \
"s_add_u32 s57, s56, 0x800000\n\t" \
"buffer_load_dwordx4 v[172:175], %[woff], s[36:39], s57 offen nt\n\t" \
"s_add_u32 s58, s56, 0x820000\n\t" \
"buffer_load_dwordx4 v[176:179], %[woff], s[36:39], s58 offen nt\n\t" \
"s_add_u32 s57, s56, 0x840000\n\t" \
"buffer_load_dwordx4 v[180:183], %[woff], s[36:39], s57 offen nt\n\t" \
"s_add_u32 s58, s56, 0x860000\n\t" \
"buffer_load_dwordx4 v[184:187], %[woff], s[36:39], s58 offen nt\n\t" \
"s_add_u32 s57, s56, 0xa00000\n\t" \
"buffer_load_dwordx4 v[188:191], %[woff], s[36:39], s57 offen nt\n\t" \
"s_add_u32 s58, s56, 0xa20000\n\t" \
"buffer_load_dwordx4 v[192:195], %[woff], s[36:39], s58 offen nt\n\t" \
"s_add_u32 s57, s56, 0xa40000\n\t" \
"buffer_load_dwordx4 v[196:199], %[woff], s[36:39], s57 offen nt\n\t" \
"s_add_u32 s58, s56, 0xa60000\n\t" \
"buffer_load_dwordx4 v[200:203], %[woff], s[36:39], s58 offen nt\n\t" \
"s_branch PDONE_%=\n\t" \
"STAGE_%=:\n\t" \
"s_add_u32 s57, s56, 0x0\n\t" \
"buffer_load_dwordx4 v[140:143], %[woff], s[36:39], s57 offen nt\n\t" \
"s_add_u32 s58, s56, 0x20000\n\t" \
"buffer_load_dwordx4 v[144:147], %[woff], s[36:39], s58 offen nt\n\t" \
"s_add_u32 s57, s56, 0x40000\n\t" \
"buffer_load_dwordx4 v[148:151], %[woff], s[36:39], s57 offen nt\n\t" \
"s_add_u32 s58, s56, 0x60000\n\t" \
"buffer_load_dwordx4 v[152:155], %[woff], s[36:39], s58 offen nt\n\t" \
"s_add_u32 s57, s56, 0x200000\n\t" \
"buffer_load_dwordx4 v[156:159], %[woff], s[36:39], s57 offen nt\n\t" \
"s_add_u32 s58, s56, 0x220000\n\t" \
"buffer_load_dwordx4 v[160:163], %[woff], s[36:39], s58 offen nt\n\t" \
"s_add_u32 s57, s56, 0x240000\n\t" \
"buffer_load_dwordx4 v[164:167], %[woff], s[36:39], s57 offen nt\n\t" \
"s_add_u32 s58, s56, 0x260000\n\t" \
"buffer_load_dwordx4 v[168:171], %[woff], s[36:39], s58 offen nt\n\t" \
"s_add_u32 s57, s56, 0x800000\n\t" \
"buffer_load_dwordx4 v[172:175], %[woff], s[36:39], s57 offen nt\n\t" \
"s_add_u32 s58, s56, 0x820000\n\t" \
"buffer_load_dwordx4 v[176:179], %[woff], s[36:39], s58 offen nt\n\t" \
"s_add_u32 s57, s56, 0x840000\n\t" \
"buffer_load_dwordx4 v[180:183], %[woff], s[36:39], s57 offen nt\n\t" \
"s_add_u32 s58, s56, 0x860000\n\t" \
"buffer_load_dwordx4 v[184:187], %[woff], s[36:39], s58 offen nt\n\t" \
"s_add_u32 s57, s56, 0xa00000\n\t" \
"buffer_load_dwordx4 v[188:191], %[woff], s[36:39], s57 offen nt\n\t" \
"s_add_u32 s58, s56, 0xa20000\n\t" \
"buffer_load_dwordx4 v[192:195], %[woff], s[36:39], s58 offen nt\n\t" \
"s_add_u32 s57, s56, 0xa40000\n\t" \
"buffer_load_dwordx4 v[196:199], %[woff], s[36:39], s57 offen nt\n\t" \
"s_add_u32 s58, s56, 0xa60000\n\t" \
"buffer_load_dwordx4 v[200:203], %[woff], s[36:39], s58 offen nt\n\t" \
"v_mfma_f32_32x32x16_f16 a[0:15], v[204:207], v[204:207], 0\n\t" \
"v_mfma_f32_32x32x16_f16 a[16:31], v[204:207], v[204:207], 0\n\t" \
"v_mfma_f32_32x32x16_f16 a[32:47], v[204:207], v[204:207], 0\n\t" \
"v_mfma_f32_32x32x16_f16 a[48:63], v[204:207], v[204:207], 0\n\t" \
"v_mfma_f32_32x32x16_f16 a[64:79], v[204:207], v[204:207], 0\n\t" \
"v_mfma_f32_32x32x16_f16 a[80:95], v[204:207], v[204:207], 0\n\t" \
"v_mfma_f32_32x32x16_f16 a[96:111], v[204:207], v[204:207], 0\n\t" \
"v_mfma_f32_32x32x16_f16 a[112:127], v[204:207], v[204:207], 0\n\t" \
"v_mfma_f32_32x32x16_f16 a[128:143], v[204:207], v[204:207], 0\n\t" \
"v_mfma_f32_32x32x16_f16 a[144:159], v[204:207], v[204:207], 0\n\t" \
"v_mfma_f32_32x32x16_f16 a[160:175], v[204:207], v[204:207], 0\n\t" \
"v_mfma_f32_32x32x16_f16 a[176:191], v[204:207], v[204:207], 0\n\t" \
"v_mfma_f32_32x32x16_f16 a[192:207], v[204:207], v[204:207], 0\n\t" \
"v_mfma_f32_32x32x16_f16 a[208:223], v[204:207], v[204:207], 0\n\t" \
"v_mfma_f32_32x32x16_f16 a[224:239], v[204:207], v[204:207], 0\n\t" \
"v_mfma_f32_32x32x16_f16 a[240:255], v[204:207], v[204:207], 0\n\t" \
"s_waitcnt vmcnt(16) lgkmcnt(0)\n\t" \
"v_readfirstlane_b32 s68, v208\n\t" \
"v_readfirstlane_b32 s66, v209\n\t" \
"v_cvt_pk_f16_f32 v44, v12, v13\n\t" \
"v_cvt_pk_f16_f32 v45, v14, v15\n\t" \
"v_cvt_pk_f16_f32 v46, v16, v17\n\t" \
"v_cvt_pk_f16_f32 v47, v18, v19\n\t" \
"v_cvt_pk_f16_f32 v48, v20, v21\n\t" \
"v_cvt_pk_f16_f32 v49, v22, v23\n\t" \
"v_cvt_pk_f16_f32 v50, v24, v25\n\t" \
"v_cvt_pk_f16_f32 v51, v26, v27\n\t" \
"v_cvt_pk_f16_f32 v52, v28, v29\n\t" \
"v_cvt_pk_f16_f32 v53, v30, v31\n\t" \
"v_cvt_pk_f16_f32 v54, v32, v33\n\t" \
"v_cvt_pk_f16_f32 v55, v34, v35\n\t" \
"v_cvt_pk_f16_f32 v56, v36, v37\n\t" \
"v_cvt_pk_f16_f32 v57, v38, v39\n\t" \
"v_cvt_pk_f16_f32 v58, v40, v41\n\t" \
"v_cvt_pk_f16_f32 v59, v42, v43\n\t" \
"s_sub_u32 s66, s70, s66\n\t" \
"s_cmp_lt_u32 s66, 6000\n\t" \
"s_cselect_b32 s89, 0xf0000000, s89\n\t" \
"s_cselect_b32 s91, 0, s91\n\t" \
"s_add_u32 s68, s68, 1\n\t" \
"s_and_b32 s63, s68, 1\n\t" \
"s_lshl_b32 s63, s63, 22\n\t" \
"s_add_u32 s63, s63, 65536\n\t" \
"s_add_u32 s78, %[dstoff], s63\n\t" \
"s_add_u32 s40, s40, s63\n\t" \
"s_addc_u32 s41, s41, 0\n\t" \
"ds_write_b128 %[ldsstg], v[44:47] offset:0\n\t" \
"ds_write_b128 %[ldsstg], v[48:51] offset:4096\n\t" \
"ds_write_b128 %[ldsstg], v[52:55] offset:8192\n\t" \
"ds_write_b128 %[ldsstg], v[56:59] offset:12288\n\t" \
"PDONE_%=:\n\t" \
"s_lshl_b32 s67, s68, 3\n\t" \
"s_lshl_b32 s87, s68, 6\n\t" \
"s_mov_b32 s74, %[pushc]\n\t" \
"s_mov_b32 s75, %[flagc]\n\t" \
"v_mbcnt_lo_u32_b32 v254, -1, 0\n\t" \
"v_mbcnt_hi_u32_b32 v254, -1, v254\n\t" \
"v_add_u32 v253, s87, v254\n\t" \
"s_cmp_eq_u32 %[wave], 0\n\t" \
"s_cselect_b32 s63, 4, 64\n\t" \
"v_cmp_gt_u32 vcc, s63, v254\n\t" \
"s_nop 1\n\t" \
"v_cndmask_b32 v253, 0, v253, vcc\n\t" \
"s_cmp_eq_u32 %[wave], 0\n\t" \
"s_cbranch_scc1 SPIN0_%=\n\t" \
"SPIN0RET_%=:\n\t" \
"s_waitcnt lgkmcnt(0)\n\t" \
"s_barrier\n\t" \
"buffer_load_dword v250, %[poff], s[80:83], 0 offen sc1\n\t" \
"s_waitcnt vmcnt(9)\n\t" \
"v_cvt_pk_f16_f32 v236, v140, v144\n\t" \
"v_cvt_pk_f16_f32 v237, v148, v152\n\t" \
"v_cvt_pk_f16_f32 v238, v156, v160\n\t" \
"v_cvt_pk_f16_f32 v239, v164, v168\n\t" \
"ds_write_b128 v249, v[236:239] offset:0\n\t" \
"v_cvt_pk_f16_f32 v240, v141, v145\n\t" \
"v_cvt_pk_f16_f32 v241, v149, v153\n\t" \
"v_cvt_pk_f16_f32 v242, v157, v161\n\t" \
"v_cvt_pk_f16_f32 v243, v165, v169\n\t" \
"ds_write_b128 v249, v[240:243] offset:1024\n\t" \
"v_cvt_pk_f16_f32 v236, v142, v146\n\t" \
"v_cvt_pk_f16_f32 v237, v150, v154\n\t" \
"v_cvt_pk_f16_f32 v238, v158, v162\n\t" \
"v_cvt_pk_f16_f32 v239, v166, v170\n\t" \
"ds_write_b128 v249, v[236:239] offset:2048\n\t" \
"v_cvt_pk_f16_f32 v240, v143, v147\n\t" \
"v_cvt_pk_f16_f32 v241, v151, v155\n\t" \
"v_cvt_pk_f16_f32 v242, v159, v163\n\t" \
"v_cvt_pk_f16_f32 v243, v167, v171\n\t" \
"ds_write_b128 v249, v[240:243] offset:3072\n\t" \
"s_mov_b32 s59, s61\n\t" \
"buffer_load_dwordx4 v[12:15], %[xoff], s[40:43], s59 offen offset:0 \n\t" \
"buffer_load_dwordx4 v[44:47], %[xoff], s[40:43], s59 offen offset:1024 \n\t" \
"buffer_load_dwordx4 v[16:19], %[xoff], s[40:43], s59 offen offset:2048 \n\t" \
"s_waitcnt vmcnt(4)\n\t" \
"v_cvt_pk_f16_f32 v236, v172, v176\n\t" \
"v_cvt_pk_f16_f32 v237, v180, v184\n\t" \
"v_cvt_pk_f16_f32 v238, v188, v192\n\t" \
"buffer_load_dwordx4 v[48:51], %[xoff], s[40:43], s59 offen offset:3072 \n\t" \
"s_add_u32 s59, s59, 4096\n\t" \
"v_cvt_pk_f16_f32 v239, v196, v200\n\t" \
"ds_write_b128 v249, v[236:239] offset:256\n\t" \
"buffer_load_dwordx4 v[20:23], %[xoff], s[40:43], s59 offen offset:0 \n\t" \
"v_cvt_pk_f16_f32 v240, v173, v177\n\t" \
"v_cvt_pk_f16_f32 v241, v181, v185\n\t" \
"v_cvt_pk_f16_f32 v242, v189, v193\n\t" \
"buffer_load_dwordx4 v[52:55], %[xoff], s[40:43], s59 offen offset:1024 \n\t" \
"v_cvt_pk_f16_f32 v243, v197, v201\n\t" \
"ds_write_b128 v249, v[240:243] offset:1280\n\t" \
"v_cvt_pk_f16_f32 v236, v174, v178\n\t" \
"v_cvt_pk_f16_f32 v237, v182, v186\n\t" \
"buffer_load_dwordx4 v[24:27], %[xoff], s[40:43], s59 offen offset:2048 \n\t" \
"v_cvt_pk_f16_f32 v238, v190, v194\n\t" \
"v_cvt_pk_f16_f32 v239, v198, v202\n\t" \
"ds_write_b128 v249, v[236:239] offset:2304\n\t" \
"buffer_load_dwordx4 v[56:59], %[xoff], s[40:43], s59 offen offset:3072 \n\t" \
"s_add_u32 s59, s59, 4096\n\t" \
"v_cvt_pk_f16_f32 v240, v175, v179\n\t" \
"v_cvt_pk_f16_f32 v241, v183, v187\n\t" \
"buffer_load_dwordx4 v[28:31], %[xoff], s[40:43], s59 offen offset:0 \n\t" \
"v_cvt_pk_f16_f32 v242, v191, v195\n\t" \
"v_cvt_pk_f16_f32 v243, v199, v203\n\t" \
"ds_write_b128 v249, v[240:243] offset:3328\n\t" \
"buffer_load_dwordx4 v[60:63], %[xoff], s[40:43], s59 offen offset:1024 \n\t" \
"buffer_load_dwordx4 v[32:35], %[xoff], s[40:43], s59 offen offset:2048 \n\t" \
"buffer_load_dwordx4 v[64:67], %[xoff], s[40:43], s59 offen offset:3072 \n\t" \
"s_add_u32 s59, s59, 4096\n\t" \
"buffer_load_dwordx4 v[36:39], %[xoff], s[40:43], s59 offen offset:0 \n\t" \
"buffer_load_dwordx4 v[68:71], %[xoff], s[40:43], s59 offen offset:1024 \n\t" \
"buffer_load_dwordx4 v[40:43], %[xoff], s[40:43], s59 offen offset:2048 \n\t" \
"s_waitcnt lgkmcnt(0)\n\t" \
"s_barrier\n\t" \
"buffer_load_dwordx4 v[72:75], %[xoff], s[40:43], s59 offen offset:3072 \n\t" \
"s_add_u32 s59, s59, 4096\n\t" \
"s_add_u32 s56, s56, 0x1000000\n\t" \
"s_add_u32 s61, s61, 0x20000\n\t" \
"buffer_load_dwordx4 v[244:247], %[pfoff], s[40:43], s61 offen\n\t" \
"s_add_u32 s57, s56, 0x0\n\t" \
"buffer_load_dwordx4 v[140:143], %[woff], s[36:39], s57 offen nt\n\t" \
"s_add_u32 s58, s56, 0x20000\n\t" \
"buffer_load_dwordx4 v[144:147], %[woff], s[36:39], s58 offen nt\n\t" \
"s_add_u32 s57, s56, 0x40000\n\t" \
"buffer_load_dwordx4 v[148:151], %[woff], s[36:39], s57 offen nt\n\t" \
"s_add_u32 s58, s56, 0x60000\n\t" \
"buffer_load_dwordx4 v[152:155], %[woff], s[36:39], s58 offen nt\n\t" \
"s_add_u32 s57, s56, 0x200000\n\t" \
"buffer_load_dwordx4 v[156:159], %[woff], s[36:39], s57 offen nt\n\t" \
"s_add_u32 s58, s56, 0x220000\n\t" \
"buffer_load_dwordx4 v[160:163], %[woff], s[36:39], s58 offen nt\n\t" \
"s_add_u32 s57, s56, 0x240000\n\t" \
"buffer_load_dwordx4 v[164:167], %[woff], s[36:39], s57 offen nt\n\t" \
"v_add_u32 v248, 0x8000, v248\n\t" \
"v_and_b32 v248, 0x1ffff, v248\n\t" \
"v_add_u32 v249, 0x8000, v249\n\t" \
"s_add_u32 s58, s56, 0x260000\n\t" \
"v_and_b32 v249, 0x1ffff, v249\n\t" \
"buffer_load_dwordx4 v[168:171], %[woff], s[36:39], s58 offen nt\n\t" \
"ds_read_b128 v[204:207], v248 offset:0\n\t" \
"s_add_u32 s57, s56, 0x800000\n\t" \
"buffer_load_dwordx4 v[172:175], %[woff], s[36:39], s57 offen nt\n\t" \
"ds_read_b128 v[208:211], v248 offset:8192\n\t" \
"ds_read_b128 v[212:215], v248 offset:1024\n\t" \
"s_add_u32 s58, s56, 0x820000\n\t" \
"buffer_load_dwordx4 v[176:179], %[woff], s[36:39], s58 offen nt\n\t" \
"ds_read_b128 v[216:219], v248 offset:9216\n\t" \
"s_add_u32 s57, s56, 0x840000\n\t" \
"buffer_load_dwordx4 v[180:183], %[woff], s[36:39], s57 offen nt\n\t" \
"s_add_u32 s58, s56, 0x860000\n\t" \
"buffer_load_dwordx4 v[184:187], %[woff], s[36:39], s58 offen nt\n\t" \
"s_nop 1\n\t" \
"GLOOP_%=:\n\t" \
"s_waitcnt lgkmcnt(0)\n\t" \
"s_waitcnt vmcnt(25)\n\t" \
"v_mfma_f32_16x16x32_f16 a[0:3], v[12:15], v[204:207], a[0:3]\n\t" \
"s_add_u32 s57, s56, 0xa00000\n\t" \
"v_mfma_f32_16x16x32_f16 a[0:3], v[44:47], v[208:211], a[0:3]\n\t" \
"buffer_load_dwordx4 v[188:191], %[woff], s[36:39], s57 offen nt\n\t" \
"v_mfma_f32_16x16x32_f16 a[4:7], v[44:47], v[216:219], a[4:7]\n\t" \
"ds_read_b128 v[220:223], v248 offset:2048\n\t" \
"v_mfma_f32_16x16x32_f16 a[4:7], v[12:15], v[212:215], a[4:7]\n\t" \
"v_mfma_f32_16x16x32_f16 a[36:39], v[16:19], v[212:215], a[36:39]\n\t" \
"v_mfma_f32_16x16x32_f16 a[36:39], v[48:51], v[216:219], a[36:39]\n\t" \
"v_mfma_f32_16x16x32_f16 a[32:35], v[48:51], v[208:211], a[32:35]\n\t" \
"ds_read_b128 v[224:227], v248 offset:10240\n\t" \
"v_mfma_f32_16x16x32_f16 a[32:35], v[16:19], v[204:207], a[32:35]\n\t" \
"s_add_u32 s58, s56, 0xa20000\n\t" \
"s_waitcnt vmcnt(22)\n\t" \
"v_mfma_f32_16x16x32_f16 a[64:67], v[20:23], v[204:207], a[64:67]\n\t" \
"buffer_load_dwordx4 v[192:195], %[woff], s[36:39], s58 offen nt\n\t" \
"v_mfma_f32_16x16x32_f16 a[64:67], v[52:55], v[208:211], a[64:67]\n\t" \
"v_mfma_f32_16x16x32_f16 a[68:71], v[52:55], v[216:219], a[68:71]\n\t" \
"ds_read_b128 v[228:231], v248 offset:3072\n\t" \
"v_mfma_f32_16x16x32_f16 a[68:71], v[20:23], v[212:215], a[68:71]\n\t" \
"v_mfma_f32_16x16x32_f16 a[100:103], v[24:27], v[212:215], a[100:103]\n\t" \
"v_mfma_f32_16x16x32_f16 a[100:103], v[56:59], v[216:219], a[100:103]\n\t" \
"s_cmp_le_i32 s91, 0\n\t" \
"s_cbranch_scc1 PC0_%=\n\t" \
"s_sleep 10\n\t" \
"PC0_%=:\n\t" \
"v_mfma_f32_16x16x32_f16 a[96:99], v[56:59], v[208:211], a[96:99]\n\t" \
"ds_read_b128 v[232:235], v248 offset:11264\n\t" \
"s_add_u32 s57, s56, 0xa40000\n\t" \
"v_mfma_f32_16x16x32_f16 a[96:99], v[24:27], v[204:207], a[96:99]\n\t" \
"buffer_load_dwordx4 v[196:199], %[woff], s[36:39], s57 offen nt\n\t" \
"s_waitcnt vmcnt(20)\n\t" \
"v_mfma_f32_16x16x32_f16 a[128:131], v[28:31], v[204:207], a[128:131]\n\t" \
"v_mfma_f32_16x16x32_f16 a[128:131], v[60:63], v[208:211], a[128:131]\n\t" \
"v_mfma_f32_16x16x32_f16 a[132:135], v[60:63], v[216:219], a[132:135]\n\t" \
"v_mfma_f32_16x16x32_f16 a[132:135], v[28:31], v[212:215], a[132:135]\n\t" \
"v_mfma_f32_16x16x32_f16 a[164:167], v[32:35], v[212:215], a[164:167]\n\t" \
"v_mfma_f32_16x16x32_f16 a[164:167], v[64:67], v[216:219], a[164:167]\n\t" \
"s_add_u32 s58, s56, 0xa60000\n\t" \
"v_mfma_f32_16x16x32_f16 a[160:163], v[64:67], v[208:211], a[160:163]\n\t" \
"buffer_load_dwordx4 v[200:203], %[woff], s[36:39], s58 offen nt\n\t" \
"v_mfma_f32_16x16x32_f16 a[160:163], v[32:35], v[204:207], a[160:163]\n\t" \
"s_waitcnt vmcnt(17)\n\t" \
"v_mfma_f32_16x16x32_f16 a[192:195], v[36:39], v[204:207], a[192:195]\n\t" \
"v_mfma_f32_16x16x32_f16 a[192:195], v[68:71], v[208:211], a[192:195]\n\t" \
"v_mfma_f32_16x16x32_f16 a[196:199], v[68:71], v[216:219], a[196:199]\n\t" \
"v_mfma_f32_16x16x32_f16 a[196:199], v[36:39], v[212:215], a[196:199]\n\t" \
"v_mfma_f32_16x16x32_f16 a[228:231], v[40:43], v[212:215], a[228:231]\n\t" \
"v_mfma_f32_16x16x32_f16 a[228:231], v[72:75], v[216:219], a[228:231]\n\t" \
"buffer_load_dword v250, %[poff], s[80:83], 0 offen sc1\n\t" \
"v_mfma_f32_16x16x32_f16 a[224:227], v[72:75], v[208:211], a[224:227]\n\t" \
"v_mfma_f32_16x16x32_f16 a[224:227], v[40:43], v[204:207], a[224:227]\n\t" \
"s_waitcnt lgkmcnt(0)\n\t" \
"v_mfma_f32_16x16x32_f16 a[8:11], v[12:15], v[220:223], a[8:11]\n\t" \
"buffer_load_dwordx4 v[76:79], %[xoff], s[40:43], s59 offen offset:0 \n\t" \
"v_mfma_f32_16x16x32_f16 a[8:11], v[44:47], v[224:227], a[8:11]\n\t" \
"v_mfma_f32_16x16x32_f16 a[12:15], v[44:47], v[232:235], a[12:15]\n\t" \
"ds_read_b128 v[204:207], v248 offset:4096\n\t" \
"v_mfma_f32_16x16x32_f16 a[12:15], v[12:15], v[228:231], a[12:15]\n\t" \
"v_mfma_f32_16x16x32_f16 a[44:47], v[16:19], v[228:231], a[44:47]\n\t" \
"buffer_load_dwordx4 v[108:111], %[xoff], s[40:43], s59 offen offset:1024 \n\t" \
"v_mfma_f32_16x16x32_f16 a[44:47], v[48:51], v[232:235], a[44:47]\n\t" \
"v_mfma_f32_16x16x32_f16 a[40:43], v[48:51], v[224:227], a[40:43]\n\t" \
"ds_read_b128 v[208:211], v248 offset:12288\n\t" \
"v_mfma_f32_16x16x32_f16 a[40:43], v[16:19], v[220:223], a[40:43]\n\t" \
"v_mfma_f32_16x16x32_f16 a[72:75], v[20:23], v[220:223], a[72:75]\n\t" \
"buffer_load_dwordx4 v[80:83], %[xoff], s[40:43], s59 offen offset:2048 \n\t" \
"v_mfma_f32_16x16x32_f16 a[72:75], v[52:55], v[224:227], a[72:75]\n\t" \
"v_mfma_f32_16x16x32_f16 a[76:79], v[52:55], v[232:235], a[76:79]\n\t" \
"ds_read_b128 v[212:215], v248 offset:5120\n\t" \
"v_mfma_f32_16x16x32_f16 a[76:79], v[20:23], v[228:231], a[76:79]\n\t" \
"v_mfma_f32_16x16x32_f16 a[108:111], v[24:27], v[228:231], a[108:111]\n\t" \
"v_mfma_f32_16x16x32_f16 a[108:111], v[56:59], v[232:235], a[108:111]\n\t" \
"s_cmp_le_i32 s91, 4\n\t" \
"s_cbranch_scc1 PC1_%=\n\t" \
"s_sleep 10\n\t" \
"PC1_%=:\n\t" \
"buffer_load_dwordx4 v[112:115], %[xoff], s[40:43], s59 offen offset:3072 \n\t" \
"v_mfma_f32_16x16x32_f16 a[104:107], v[56:59], v[224:227], a[104:107]\n\t" \
"ds_read_b128 v[216:219], v248 offset:13312\n\t" \
"s_add_u32 s59, s59, 4096\n\t" \
"v_mfma_f32_16x16x32_f16 a[104:107], v[24:27], v[220:223], a[104:107]\n\t" \
"v_mfma_f32_16x16x32_f16 a[136:139], v[28:31], v[220:223], a[136:139]\n\t" \
"v_mfma_f32_16x16x32_f16 a[136:139], v[60:63], v[224:227], a[136:139]\n\t" \
"buffer_load_dwordx4 v[84:87], %[xoff], s[40:43], s59 offen offset:0 \n\t" \
"v_mfma_f32_16x16x32_f16 a[140:143], v[60:63], v[232:235], a[140:143]\n\t" \
"v_mfma_f32_16x16x32_f16 a[140:143], v[28:31], v[228:231], a[140:143]\n\t" \
"v_mfma_f32_16x16x32_f16 a[172:175], v[32:35], v[228:231], a[172:175]\n\t" \
"v_mfma_f32_16x16x32_f16 a[172:175], v[64:67], v[232:235], a[172:175]\n\t" \
"buffer_load_dwordx4 v[116:119], %[xoff], s[40:43], s59 offen offset:1024 \n\t" \
"v_mfma_f32_16x16x32_f16 a[168:171], v[64:67], v[224:227], a[168:171]\n\t" \
"v_mfma_f32_16x16x32_f16 a[168:171], v[32:35], v[220:223], a[168:171]\n\t" \
"v_mfma_f32_16x16x32_f16 a[200:203], v[36:39], v[220:223], a[200:203]\n\t" \
"v_mfma_f32_16x16x32_f16 a[200:203], v[68:71], v[224:227], a[200:203]\n\t" \
"v_mfma_f32_16x16x32_f16 a[204:207], v[68:71], v[232:235], a[204:207]\n\t" \
"buffer_load_dwordx4 v[88:91], %[xoff], s[40:43], s59 offen offset:2048 \n\t" \
"v_mfma_f32_16x16x32_f16 a[204:207], v[36:39], v[228:231], a[204:207]\n\t" \
"v_mfma_f32_16x16x32_f16 a[236:239], v[40:43], v[228:231], a[236:239]\n\t" \
"v_mfma_f32_16x16x32_f16 a[236:239], v[72:75], v[232:235], a[236:239]\n\t" \
"v_mfma_f32_16x16x32_f16 a[232:235], v[72:75], v[224:227], a[232:235]\n\t" \
"buffer_load_dwordx4 v[120:123], %[xoff], s[40:43], s59 offen offset:3072 \n\t" \
"v_mfma_f32_16x16x32_f16 a[232:235], v[40:43], v[220:223], a[232:235]\n\t" \
"s_add_u32 s59, s59, 4096\n\t" \
"s_waitcnt lgkmcnt(0)\n\t" \
"v_mfma_f32_16x16x32_f16 a[16:19], v[12:15], v[204:207], a[16:19]\n\t" \
"s_waitcnt vmcnt(17)\n\t" \
"v_mfma_f32_16x16x32_f16 a[16:19], v[44:47], v[208:211], a[16:19]\n\t" \
"v_cvt_pk_f16_f32 v236, v140, v144\n\t" \
"v_mfma_f32_16x16x32_f16 a[20:23], v[44:47], v[216:219], a[20:23]\n\t" \
"ds_read_b128 v[220:223], v248 offset:6144\n\t" \
"buffer_load_dwordx4 v[92:95], %[xoff], s[40:43], s59 offen offset:0 \n\t" \
"v_mfma_f32_16x16x32_f16 a[20:23], v[12:15], v[212:215], a[20:23]\n\t" \
"v_cvt_pk_f16_f32 v237, v148, v152\n\t" \
"v_mfma_f32_16x16x32_f16 a[52:55], v[16:19], v[212:215], a[52:55]\n\t" \
"v_cvt_pk_f16_f32 v238, v156, v160\n\t" \
"v_mfma_f32_16x16x32_f16 a[52:55], v[48:51], v[216:219], a[52:55]\n\t" \
"v_cvt_pk_f16_f32 v239, v164, v168\n\t" \
"v_mfma_f32_16x16x32_f16 a[48:51], v[48:51], v[208:211], a[48:51]\n\t" \
"ds_read_b128 v[224:227], v248 offset:14336\n\t" \
"buffer_load_dwordx4 v[124:127], %[xoff], s[40:43], s59 offen offset:1024 \n\t" \
"v_mfma_f32_16x16x32_f16 a[48:51], v[16:19], v[204:207], a[48:51]\n\t" \
"ds_write_b128 v249, v[236:239] offset:0\n\t" \
"v_mfma_f32_16x16x32_f16 a[80:83], v[20:23], v[204:207], a[80:83]\n\t" \
"v_cvt_pk_f16_f32 v240, v141, v145\n\t" \
"v_mfma_f32_16x16x32_f16 a[80:83], v[52:55], v[208:211], a[80:83]\n\t" \
"v_cvt_pk_f16_f32 v241, v149, v153\n\t" \
"v_mfma_f32_16x16x32_f16 a[84:87], v[52:55], v[216:219], a[84:87]\n\t" \
"ds_read_b128 v[228:231], v248 offset:7168\n\t" \
"buffer_load_dwordx4 v[96:99], %[xoff], s[40:43], s59 offen offset:2048 \n\t" \
"v_mfma_f32_16x16x32_f16 a[84:87], v[20:23], v[212:215], a[84:87]\n\t" \
"v_cvt_pk_f16_f32 v242, v157, v161\n\t" \
"v_mfma_f32_16x16x32_f16 a[116:119], v[24:27], v[212:215], a[116:119]\n\t" \
"v_cvt_pk_f16_f32 v243, v165, v169\n\t" \
"v_mfma_f32_16x16x32_f16 a[116:119], v[56:59], v[216:219], a[116:119]\n\t" \
"s_cmp_le_i32 s91, 2\n\t" \
"s_cbranch_scc1 PC2_%=\n\t" \
"s_sleep 10\n\t" \
"PC2_%=:\n\t" \
"v_mfma_f32_16x16x32_f16 a[112:115], v[56:59], v[208:211], a[112:115]\n\t" \
"ds_read_b128 v[232:235], v248 offset:15360\n\t" \
"v_mfma_f32_16x16x32_f16 a[112:115], v[24:27], v[204:207], a[112:115]\n\t" \
"buffer_load_dwordx4 v[128:131], %[xoff], s[40:43], s59 offen offset:3072 \n\t" \
"v_mfma_f32_16x16x32_f16 a[144:147], v[28:31], v[204:207], a[144:147]\n\t" \
"s_add_u32 s59, s59, 4096\n\t" \
"v_mfma_f32_16x16x32_f16 a[144:147], v[60:63], v[208:211], a[144:147]\n\t" \
"ds_write_b128 v249, v[240:243] offset:1024\n\t" \
"v_mfma_f32_16x16x32_f16 a[148:151], v[60:63], v[216:219], a[148:151]\n\t" \
"v_cvt_pk_f16_f32 v236, v142, v146\n\t" \
"v_mfma_f32_16x16x32_f16 a[148:151], v[28:31], v[212:215], a[148:151]\n\t" \
"buffer_load_dwordx4 v[100:103], %[xoff], s[40:43], s59 offen offset:0 \n\t" \
"v_mfma_f32_16x16x32_f16 a[180:183], v[32:35], v[212:215], a[180:183]\n\t" \
"v_cvt_pk_f16_f32 v237, v150, v154\n\t" \
"v_mfma_f32_16x16x32_f16 a[180:183], v[64:67], v[216:219], a[180:183]\n\t" \
"v_cvt_pk_f16_f32 v238, v158, v162\n\t" \
"v_mfma_f32_16x16x32_f16 a[176:179], v[64:67], v[208:211], a[176:179]\n\t" \
"v_cvt_pk_f16_f32 v239, v166, v170\n\t" \
"v_mfma_f32_16x16x32_f16 a[176:179], v[32:35], v[204:207], a[176:179]\n\t" \
"buffer_load_dwordx4 v[132:135], %[xoff], s[40:43], s59 offen offset:1024 \n\t" \
"v_mfma_f32_16x16x32_f16 a[208:211], v[36:39], v[204:207], a[208:211]\n\t" \
"ds_write_b128 v249, v[236:239] offset:2048\n\t" \
"v_mfma_f32_16x16x32_f16 a[208:211], v[68:71], v[208:211], a[208:211]\n\t" \
"v_cvt_pk_f16_f32 v240, v143, v147\n\t" \
"v_mfma_f32_16x16x32_f16 a[212:215], v[68:71], v[216:219], a[212:215]\n\t" \
"v_cvt_pk_f16_f32 v241, v151, v155\n\t" \
"v_mfma_f32_16x16x32_f16 a[212:215], v[36:39], v[212:215], a[212:215]\n\t" \
"v_cvt_pk_f16_f32 v242, v159, v163\n\t" \
"v_mfma_f32_16x16x32_f16 a[244:247], v[40:43], v[212:215], a[244:247]\n\t" \
"buffer_load_dwordx4 v[104:107], %[xoff], s[40:43], s59 offen offset:2048 \n\t" \
"v_mfma_f32_16x16x32_f16 a[244:247], v[72:75], v[216:219], a[244:247]\n\t" \
"v_cvt_pk_f16_f32 v243, v167, v171\n\t" \
"v_mfma_f32_16x16x32_f16 a[240:243], v[72:75], v[208:211], a[240:243]\n\t" \
"ds_write_b128 v249, v[240:243] offset:3072\n\t" \
"v_mfma_f32_16x16x32_f16 a[240:243], v[40:43], v[204:207], a[240:243]\n\t" \
"s_waitcnt lgkmcnt(3)\n\t" \
"v_mfma_f32_16x16x32_f16 a[24:27], v[12:15], v[220:223], a[24:27]\n\t" \
"buffer_load_dwordx4 v[136:139], %[xoff], s[40:43], s59 offen offset:3072 \n\t" \
"v_mfma_f32_16x16x32_f16 a[24:27], v[44:47], v[224:227], a[24:27]\n\t" \
"v_mfma_f32_16x16x32_f16 a[28:31], v[44:47], v[232:235], a[28:31]\n\t" \
"ds_read_b128 v[204:207], v248 offset:16384\n\t" \
"v_mfma_f32_16x16x32_f16 a[28:31], v[12:15], v[228:231], a[28:31]\n\t" \
"s_mov_b32 s59, s61\n\t" \
"v_mfma_f32_16x16x32_f16 a[60:63], v[16:19], v[228:231], a[60:63]\n\t" \
"buffer_load_dwordx4 v[12:15], %[xoff], s[40:43], s59 offen offset:0 \n\t" \
"v_mfma_f32_16x16x32_f16 a[60:63], v[48:51], v[232:235], a[60:63]\n\t" \
"v_mfma_f32_16x16x32_f16 a[56:59], v[48:51], v[224:227], a[56:59]\n\t" \
"ds_read_b128 v[208:211], v248 offset:24576\n\t" \
"v_mfma_f32_16x16x32_f16 a[56:59], v[16:19], v[220:223], a[56:59]\n\t" \
"v_mfma_f32_16x16x32_f16 a[88:91], v[20:23], v[220:223], a[88:91]\n\t" \
"buffer_load_dwordx4 v[44:47], %[xoff], s[40:43], s59 offen offset:1024 \n\t" \
"v_mfma_f32_16x16x32_f16 a[88:91], v[52:55], v[224:227], a[88:91]\n\t" \
"v_mfma_f32_16x16x32_f16 a[92:95], v[52:55], v[232:235], a[92:95]\n\t" \
"ds_read_b128 v[212:215], v248 offset:17408\n\t" \
"v_mfma_f32_16x16x32_f16 a[92:95], v[20:23], v[228:231], a[92:95]\n\t" \
"v_mfma_f32_16x16x32_f16 a[124:127], v[24:27], v[228:231], a[124:127]\n\t" \
"buffer_load_dwordx4 v[16:19], %[xoff], s[40:43], s59 offen offset:2048 \n\t" \
"v_mfma_f32_16x16x32_f16 a[124:127], v[56:59], v[232:235], a[124:127]\n\t" \
"s_cmp_le_i32 s91, 6\n\t" \
"s_cbranch_scc1 PC3_%=\n\t" \
"s_sleep 10\n\t" \
"PC3_%=:\n\t" \
"v_mfma_f32_16x16x32_f16 a[120:123], v[56:59], v[224:227], a[120:123]\n\t" \
"ds_read_b128 v[216:219], v248 offset:25600\n\t" \
"v_mfma_f32_16x16x32_f16 a[120:123], v[24:27], v[220:223], a[120:123]\n\t" \
"s_waitcnt vmcnt(20)\n\t" \
"v_mfma_f32_16x16x32_f16 a[152:155], v[28:31], v[220:223], a[152:155]\n\t" \
"v_cvt_pk_f16_f32 v236, v172, v176\n\t" \
"v_mfma_f32_16x16x32_f16 a[152:155], v[60:63], v[224:227], a[152:155]\n\t" \
"buffer_load_dwordx4 v[48:51], %[xoff], s[40:43], s59 offen offset:3072 \n\t" \
"v_mfma_f32_16x16x32_f16 a[156:159], v[60:63], v[232:235], a[156:159]\n\t" \
"s_add_u32 s59, s59, 4096\n\t" \
"v_mfma_f32_16x16x32_f16 a[156:159], v[28:31], v[228:231], a[156:159]\n\t" \
"v_cvt_pk_f16_f32 v237, v180, v184\n\t" \
"v_mfma_f32_16x16x32_f16 a[188:191], v[32:35], v[228:231], a[188:191]\n\t" \
"v_cvt_pk_f16_f32 v238, v188, v192\n\t" \
"v_mfma_f32_16x16x32_f16 a[188:191], v[64:67], v[232:235], a[188:191]\n\t" \
"buffer_load_dwordx4 v[20:23], %[xoff], s[40:43], s59 offen offset:0 \n\t" \
"v_mfma_f32_16x16x32_f16 a[184:187], v[64:67], v[224:227], a[184:187]\n\t" \
"v_cvt_pk_f16_f32 v239, v196, v200\n\t" \
"v_mfma_f32_16x16x32_f16 a[184:187], v[32:35], v[220:223], a[184:187]\n\t" \
"ds_write_b128 v249, v[236:239] offset:256\n\t" \
"v_mfma_f32_16x16x32_f16 a[216:219], v[36:39], v[220:223], a[216:219]\n\t" \
"v_cvt_pk_f16_f32 v240, v173, v177\n\t" \
"v_mfma_f32_16x16x32_f16 a[216:219], v[68:71], v[224:227], a[216:219]\n\t" \
"buffer_load_dwordx4 v[52:55], %[xoff], s[40:43], s59 offen offset:1024 \n\t" \
"v_mfma_f32_16x16x32_f16 a[220:223], v[68:71], v[232:235], a[220:223]\n\t" \
"v_cvt_pk_f16_f32 v241, v181, v185\n\t" \
"v_mfma_f32_16x16x32_f16 a[220:223], v[36:39], v[228:231], a[220:223]\n\t" \
"v_cvt_pk_f16_f32 v242, v189, v193\n\t" \
"v_mfma_f32_16x16x32_f16 a[252:255], v[40:43], v[228:231], a[252:255]\n\t" \
"v_cvt_pk_f16_f32 v243, v197, v201\n\t" \
"v_mfma_f32_16x16x32_f16 a[252:255], v[72:75], v[232:235], a[252:255]\n\t" \
"ds_write_b128 v249, v[240:243] offset:1280\n\t" \
"v_mfma_f32_16x16x32_f16 a[248:251], v[72:75], v[224:227], a[248:251]\n\t" \
"buffer_load_dwordx4 v[24:27], %[xoff], s[40:43], s59 offen offset:2048 \n\t" \
"v_mfma_f32_16x16x32_f16 a[248:251], v[40:43], v[220:223], a[248:251]\n\t" \
"v_cvt_pk_f16_f32 v236, v174, v178\n\t" \
"s_waitcnt lgkmcnt(2)\n\t" \
"s_waitcnt vmcnt(19)\n\t" \
"v_mfma_f32_16x16x32_f16 a[0:3], v[76:79], v[204:207], a[0:3]\n\t" \
"v_cvt_pk_f16_f32 v237, v182, v186\n\t" \
"v_mfma_f32_16x16x32_f16 a[0:3], v[108:111], v[208:211], a[0:3]\n\t" \
"v_cvt_pk_f16_f32 v238, v190, v194\n\t" \
"v_mfma_f32_16x16x32_f16 a[4:7], v[108:111], v[216:219], a[4:7]\n\t" \
"ds_read_b128 v[220:223], v248 offset:18432\n\t" \
"buffer_load_dwordx4 v[56:59], %[xoff], s[40:43], s59 offen offset:3072 \n\t" \
"v_mfma_f32_16x16x32_f16 a[4:7], v[76:79], v[212:215], a[4:7]\n\t" \
"s_add_u32 s59, s59, 4096\n\t" \
"v_mfma_f32_16x16x32_f16 a[36:39], v[80:83], v[212:215], a[36:39]\n\t" \
"v_cvt_pk_f16_f32 v239, v198, v202\n\t" \
"v_mfma_f32_16x16x32_f16 a[36:39], v[112:115], v[216:219], a[36:39]\n\t" \
"ds_write_b128 v249, v[236:239] offset:2304\n\t" \
"v_mfma_f32_16x16x32_f16 a[32:35], v[112:115], v[208:211], a[32:35]\n\t" \
"ds_read_b128 v[224:227], v248 offset:26624\n\t" \
"buffer_load_dwordx4 v[28:31], %[xoff], s[40:43], s59 offen offset:0 \n\t" \
"v_mfma_f32_16x16x32_f16 a[32:35], v[80:83], v[204:207], a[32:35]\n\t" \
"v_cvt_pk_f16_f32 v240, v175, v179\n\t" \
"s_waitcnt vmcnt(17)\n\t" \
"v_mfma_f32_16x16x32_f16 a[64:67], v[84:87], v[204:207], a[64:67]\n\t" \
"v_cvt_pk_f16_f32 v241, v183, v187\n\t" \
"v_mfma_f32_16x16x32_f16 a[64:67], v[116:119], v[208:211], a[64:67]\n\t" \
"v_cvt_pk_f16_f32 v242, v191, v195\n\t" \
"v_mfma_f32_16x16x32_f16 a[68:71], v[116:119], v[216:219], a[68:71]\n\t" \
"ds_read_b128 v[228:231], v248 offset:19456\n\t" \
"buffer_load_dwordx4 v[60:63], %[xoff], s[40:43], s59 offen offset:1024 \n\t" \
"v_mfma_f32_16x16x32_f16 a[68:71], v[84:87], v[212:215], a[68:71]\n\t" \
"v_cvt_pk_f16_f32 v243, v199, v203\n\t" \
"v_mfma_f32_16x16x32_f16 a[100:103], v[88:91], v[212:215], a[100:103]\n\t" \
"ds_write_b128 v249, v[240:243] offset:3328\n\t" \
"v_mfma_f32_16x16x32_f16 a[100:103], v[120:123], v[216:219], a[100:103]\n\t" \
"s_cmp_le_i32 s91, 1\n\t" \
"s_cbranch_scc1 PC4_%=\n\t" \
"s_sleep 10\n\t" \
"PC4_%=:\n\t" \
"v_mfma_f32_16x16x32_f16 a[96:99], v[120:123], v[208:211], a[96:99]\n\t" \
"ds_read_b128 v[232:235], v248 offset:27648\n\t" \
"buffer_load_dwordx4 v[32:35], %[xoff], s[40:43], s59 offen offset:2048 \n\t" \
"v_mfma_f32_16x16x32_f16 a[96:99], v[88:91], v[204:207], a[96:99]\n\t" \
"s_waitcnt vmcnt(15)\n\t" \
"v_mfma_f32_16x16x32_f16 a[128:131], v[92:95], v[204:207], a[128:131]\n\t" \
"s_waitcnt vmcnt(27)\n\t" \
"v_mfma_f32_16x16x32_f16 a[128:131], v[124:127], v[208:211], a[128:131]\n\t" \
"v_max_u32 v253, v253, v250\n\t" \
"v_cmp_le_u32 vcc, s67, v250\n\t" \
"v_mfma_f32_16x16x32_f16 a[132:135], v[124:127], v[216:219], a[132:135]\n\t" \
"v_mfma_f32_16x16x32_f16 a[132:135], v[92:95], v[212:215], a[132:135]\n\t" \
"s_nop 0\n\t" \
"s_andn2_b32 s65, vcc_hi, s79\n\t" \
"s_cbranch_scc1 APPEND_%=\n\t" \
"APPRET_%=:\n\t" \
"buffer_load_dwordx4 v[64:67], %[xoff], s[40:43], s59 offen offset:3072 \n\t" \
"v_mfma_f32_16x16x32_f16 a[164:167], v[96:99], v[212:215], a[164:167]\n\t" \
"v_cmp_le_u32 vcc, s87, v253\n\t" \
"s_add_u32 s59, s59, 4096\n\t" \
"v_mfma_f32_16x16x32_f16 a[164:167], v[128:131], v[216:219], a[164:167]\n\t" \
"v_readlane_b32 s63, v253, s64\n\t" \
"v_mfma_f32_16x16x32_f16 a[160:163], v[128:131], v[208:211], a[160:163]\n\t" \
"v_mfma_f32_16x16x32_f16 a[160:163], v[96:99], v[204:207], a[160:163]\n\t" \
"s_cmp_eq_u32 vcc_lo, -1\n\t" \
"s_cselect_b32 s82, 0, s82\n\t" \
"buffer_load_dwordx4 v[36:39], %[xoff], s[40:43], s59 offen offset:0 \n\t" \
"s_waitcnt vmcnt(13)\n\t" \
"v_mfma_f32_16x16x32_f16 a[192:195], v[100:103], v[204:207], a[192:195]\n\t" \
"s_cmp_ge_u32 s63, s87\n\t" \
"s_cbranch_scc0 SPINL_%=\n\t" \
"DECRET_%=:\n\t" \
"s_and_b32 s66, s63, 31\n\t" \
"s_or_b32 s66, s66, s69\n\t" \
"v_mfma_f32_16x16x32_f16 a[192:195], v[132:135], v[208:211], a[192:195]\n\t" \
"v_mov_b32 v251, s66\n\t" \
"v_mfma_f32_16x16x32_f16 a[196:199], v[132:135], v[216:219], a[196:199]\n\t" \
"s_add_u32 s64, s64, 1\n\t" \
"s_min_u32 s64, s64, 31\n\t" \
"v_mfma_f32_16x16x32_f16 a[196:199], v[100:103], v[212:215], a[196:199]\n\t" \
"ds_write_b32 %[ldsdec], v251\n\t" \
"buffer_load_dwordx4 v[68:71], %[xoff], s[40:43], s59 offen offset:1024 \n\t" \
"v_mfma_f32_16x16x32_f16 a[228:231], v[104:107], v[212:215], a[228:231]\n\t" \
"v_mfma_f32_16x16x32_f16 a[228:231], v[136:139], v[216:219], a[228:231]\n\t" \
"v_mfma_f32_16x16x32_f16 a[224:227], v[136:139], v[208:211], a[224:227]\n\t" \
"v_mfma_f32_16x16x32_f16 a[224:227], v[104:107], v[204:207], a[224:227]\n\t" \
"s_waitcnt lgkmcnt(1)\n\t" \
"v_mfma_f32_16x16x32_f16 a[8:11], v[76:79], v[220:223], a[8:11]\n\t" \
"buffer_load_dwordx4 v[40:43], %[xoff], s[40:43], s59 offen offset:2048 \n\t" \
"v_mfma_f32_16x16x32_f16 a[8:11], v[108:111], v[224:227], a[8:11]\n\t" \
"s_waitcnt lgkmcnt(0)\n\t" \
"s_barrier\n\t" \
"v_mfma_f32_16x16x32_f16 a[12:15], v[108:111], v[232:235], a[12:15]\n\t" \
"ds_read_b32 v251, %[ldsd0]\n\t" \
"ds_read_b128 v[204:207], v248 offset:20480\n\t" \
"v_mfma_f32_16x16x32_f16 a[12:15], v[76:79], v[228:231], a[12:15]\n\t" \
"s_cmp_eq_u32 s62, s75\n\t" \
"s_cbranch_scc1 FLAGB_%=\n\t" \
"FLAGRET_%=:\n\t" \
"v_mfma_f32_16x16x32_f16 a[44:47], v[80:83], v[228:231], a[44:47]\n\t" \
"buffer_load_dwordx4 v[72:75], %[xoff], s[40:43], s59 offen offset:3072 \n\t" \
"v_mfma_f32_16x16x32_f16 a[44:47], v[112:115], v[232:235], a[44:47]\n\t" \
"s_add_u32 s59, s59, 4096\n\t" \
"v_mfma_f32_16x16x32_f16 a[40:43], v[112:115], v[224:227], a[40:43]\n\t" \
"ds_read_b128 v[208:211], v248 offset:28672\n\t" \
"v_mfma_f32_16x16x32_f16 a[40:43], v[80:83], v[220:223], a[40:43]\n\t" \
"s_waitcnt lgkmcnt(2)\n\t" \
"v_readfirstlane_b32 s66, v251\n\t" \
"v_mfma_f32_16x16x32_f16 a[72:75], v[84:87], v[220:223], a[72:75]\n\t" \
"v_mfma_f32_16x16x32_f16 a[72:75], v[116:119], v[224:227], a[72:75]\n\t" \
"s_and_b32 s69, s66, 256\n\t" \
"s_and_b32 s66, s66, 31\n\t" \
"v_readlane_b32 s63, %[uid4], s66\n\t" \
"v_mfma_f32_16x16x32_f16 a[76:79], v[116:119], v[232:235], a[76:79]\n\t" \
"ds_read_b128 v[212:215], v248 offset:21504\n\t" \
"v_mfma_f32_16x16x32_f16 a[76:79], v[84:87], v[228:231], a[76:79]\n\t" \
"s_lshl_b32 s56, s63, 22\n\t" \
"s_lshl_b32 s61, s63, 15\n\t" \
"v_mfma_f32_16x16x32_f16 a[108:111], v[88:91], v[228:231], a[108:111]\n\t" \
"s_cmp_eq_u32 s62, 2\n\t" \
"s_cselect_b32 s38, 0, s38\n\t" \
"buffer_load_dwordx4 v[244:247], %[pfoff], s[40:43], s61 offen\n\t" \
"v_mfma_f32_16x16x32_f16 a[108:111], v[120:123], v[232:235], a[108:111]\n\t" \
"s_cmp_le_i32 s91, 5\n\t" \
"s_cbranch_scc1 PC5_%=\n\t" \
"s_sleep 10\n\t" \
"PC5_%=:\n\t" \
"s_cmp_eq_u32 s62, s74\n\t" \
"s_cbranch_scc1 PUSHB_%=\n\t" \
"PUSHRET_%=:\n\t" \
"s_add_u32 s57, s56, 0x0\n\t" \
"v_mfma_f32_16x16x32_f16 a[104:107], v[120:123], v[224:227], a[104:107]\n\t" \
"ds_read_b128 v[216:219], v248 offset:29696\n\t" \
"buffer_load_dwordx4 v[140:143], %[woff], s[36:39], s57 offen nt\n\t" \
"v_mfma_f32_16x16x32_f16 a[104:107], v[88:91], v[220:223], a[104:107]\n\t" \
"v_mfma_f32_16x16x32_f16 a[136:139], v[92:95], v[220:223], a[136:139]\n\t" \
"v_mfma_f32_16x16x32_f16 a[136:139], v[124:127], v[224:227], a[136:139]\n\t" \
"v_mfma_f32_16x16x32_f16 a[140:143], v[124:127], v[232:235], a[140:143]\n\t" \
"v_mfma_f32_16x16x32_f16 a[140:143], v[92:95], v[228:231], a[140:143]\n\t" \
"v_mfma_f32_16x16x32_f16 a[172:175], v[96:99], v[228:231], a[172:175]\n\t" \
"s_add_u32 s58, s56, 0x20000\n\t" \
"v_mfma_f32_16x16x32_f16 a[172:175], v[128:131], v[232:235], a[172:175]\n\t" \
"buffer_load_dwordx4 v[144:147], %[woff], s[36:39], s58 offen nt\n\t" \
"v_mfma_f32_16x16x32_f16 a[168:171], v[128:131], v[224:227], a[168:171]\n\t" \
"v_mfma_f32_16x16x32_f16 a[168:171], v[96:99], v[220:223], a[168:171]\n\t" \
"v_mfma_f32_16x16x32_f16 a[200:203], v[100:103], v[220:223], a[200:203]\n\t" \
"v_mfma_f32_16x16x32_f16 a[200:203], v[132:135], v[224:227], a[200:203]\n\t" \
"v_mfma_f32_16x16x32_f16 a[204:207], v[132:135], v[232:235], a[204:207]\n\t" \
"v_mfma_f32_16x16x32_f16 a[204:207], v[100:103], v[228:231], a[204:207]\n\t" \
"s_add_u32 s57, s56, 0x40000\n\t" \
"v_mfma_f32_16x16x32_f16 a[236:239], v[104:107], v[228:231], a[236:239]\n\t" \
"buffer_load_dwordx4 v[148:151], %[woff], s[36:39], s57 offen nt\n\t" \
"v_mfma_f32_16x16x32_f16 a[236:239], v[136:139], v[232:235], a[236:239]\n\t" \
"v_mfma_f32_16x16x32_f16 a[232:235], v[136:139], v[224:227], a[232:235]\n\t" \
"v_mfma_f32_16x16x32_f16 a[232:235], v[104:107], v[220:223], a[232:235]\n\t" \
"s_waitcnt lgkmcnt(0)\n\t" \
"v_mfma_f32_16x16x32_f16 a[16:19], v[76:79], v[204:207], a[16:19]\n\t" \
"v_mfma_f32_16x16x32_f16 a[16:19], v[108:111], v[208:211], a[16:19]\n\t" \
"v_mfma_f32_16x16x32_f16 a[20:23], v[108:111], v[216:219], a[20:23]\n\t" \
"ds_read_b128 v[220:223], v248 offset:22528\n\t" \
"s_add_u32 s58, s56, 0x60000\n\t" \
"v_mfma_f32_16x16x32_f16 a[20:23], v[76:79], v[212:215], a[20:23]\n\t" \
"buffer_load_dwordx4 v[152:155], %[woff], s[36:39], s58 offen nt\n\t" \
"v_mfma_f32_16x16x32_f16 a[52:55], v[80:83], v[212:215], a[52:55]\n\t" \
"v_mfma_f32_16x16x32_f16 a[52:55], v[112:115], v[216:219], a[52:55]\n\t" \
"v_mfma_f32_16x16x32_f16 a[48:51], v[112:115], v[208:211], a[48:51]\n\t" \
"ds_read_b128 v[224:227], v248 offset:30720\n\t" \
"v_mfma_f32_16x16x32_f16 a[48:51], v[80:83], v[204:207], a[48:51]\n\t" \
"v_mfma_f32_16x16x32_f16 a[80:83], v[84:87], v[204:207], a[80:83]\n\t" \
"v_mfma_f32_16x16x32_f16 a[80:83], v[116:119], v[208:211], a[80:83]\n\t" \
"s_add_u32 s57, s56, 0x200000\n\t" \
"v_mfma_f32_16x16x32_f16 a[84:87], v[116:119], v[216:219], a[84:87]\n\t" \
"ds_read_b128 v[228:231], v248 offset:23552\n\t" \
"buffer_load_dwordx4 v[156:159], %[woff], s[36:39], s57 offen nt\n\t" \
"v_mfma_f32_16x16x32_f16 a[84:87], v[84:87], v[212:215], a[84:87]\n\t" \
"v_mfma_f32_16x16x32_f16 a[116:119], v[88:91], v[212:215], a[116:119]\n\t" \
"v_mfma_f32_16x16x32_f16 a[116:119], v[120:123], v[216:219], a[116:119]\n\t" \
"s_cmp_le_i32 s91, 3\n\t" \
"s_cbranch_scc1 PC6_%=\n\t" \
"s_sleep 10\n\t" \
"PC6_%=:\n\t" \
"v_mfma_f32_16x16x32_f16 a[112:115], v[120:123], v[208:211], a[112:115]\n\t" \
"ds_read_b128 v[232:235], v248 offset:31744\n\t" \
"v_mfma_f32_16x16x32_f16 a[112:115], v[88:91], v[204:207], a[112:115]\n\t" \
"s_add_u32 s58, s56, 0x220000\n\t" \
"v_mfma_f32_16x16x32_f16 a[144:147], v[92:95], v[204:207], a[144:147]\n\t" \
"buffer_load_dwordx4 v[160:163], %[woff], s[36:39], s58 offen nt\n\t" \
"v_mfma_f32_16x16x32_f16 a[144:147], v[124:127], v[208:211], a[144:147]\n\t" \
"v_mfma_f32_16x16x32_f16 a[148:151], v[124:127], v[216:219], a[148:151]\n\t" \
"s_memrealtime s[92:93]\n\t" \
"v_mfma_f32_16x16x32_f16 a[148:151], v[92:95], v[212:215], a[148:151]\n\t" \
"v_mfma_f32_16x16x32_f16 a[180:183], v[96:99], v[212:215], a[180:183]\n\t" \
"v_mfma_f32_16x16x32_f16 a[180:183], v[128:131], v[216:219], a[180:183]\n\t" \
"v_mfma_f32_16x16x32_f16 a[176:179], v[128:131], v[208:211], a[176:179]\n\t" \
"s_add_u32 s57, s56, 0x240000\n\t" \
"v_mfma_f32_16x16x32_f16 a[176:179], v[96:99], v[204:207], a[176:179]\n\t" \
"buffer_load_dwordx4 v[164:167], %[woff], s[36:39], s57 offen nt\n\t" \
"v_mfma_f32_16x16x32_f16 a[208:211], v[100:103], v[204:207], a[208:211]\n\t" \
"v_add_u32 v248, 0x8000, v248\n\t" \
"v_mfma_f32_16x16x32_f16 a[208:211], v[132:135], v[208:211], a[208:211]\n\t" \
"v_and_b32 v248, 0x1ffff, v248\n\t" \
"v_mfma_f32_16x16x32_f16 a[212:215], v[132:135], v[216:219], a[212:215]\n\t" \
"v_mfma_f32_16x16x32_f16 a[212:215], v[100:103], v[212:215], a[212:215]\n\t" \
"v_mfma_f32_16x16x32_f16 a[244:247], v[104:107], v[212:215], a[244:247]\n\t" \
"v_add_u32 v249, 0x8000, v249\n\t" \
"v_mfma_f32_16x16x32_f16 a[244:247], v[136:139], v[216:219], a[244:247]\n\t" \
"s_add_u32 s58, s56, 0x260000\n\t" \
"v_and_b32 v249, 0x1ffff, v249\n\t" \
"v_mfma_f32_16x16x32_f16 a[240:243], v[136:139], v[208:211], a[240:243]\n\t" \
"buffer_load_dwordx4 v[168:171], %[woff], s[36:39], s58 offen nt\n\t" \
"v_mfma_f32_16x16x32_f16 a[240:243], v[104:107], v[204:207], a[240:243]\n\t" \
"s_waitcnt lgkmcnt(0)\n\t" \
"s_waitcnt lgkmcnt(0)\n\t" \
"v_mfma_f32_16x16x32_f16 a[24:27], v[76:79], v[220:223], a[24:27]\n\t" \
"v_mfma_f32_16x16x32_f16 a[24:27], v[108:111], v[224:227], a[24:27]\n\t" \
"s_sub_u32 s92, s92, s70\n\t" \
"s_sub_i32 s91, s89, s92\n\t" \
"s_sub_i32 s94, s94, 170\n\t" \
"s_max_i32 s94, s94, s91\n\t" \
"v_mfma_f32_16x16x32_f16 a[28:31], v[108:111], v[232:235], a[28:31]\n\t" \
"ds_read_b128 v[204:207], v248 offset:0\n\t" \
"s_add_u32 s89, s92, s94\n\t" \
"s_ashr_i32 s91, s94, 5\n\t" \
"v_mfma_f32_16x16x32_f16 a[28:31], v[76:79], v[228:231], a[28:31]\n\t" \
"s_add_u32 s89, s89, s90\n\t" \
"s_cmp_le_u32 s62, 28\n\t" \
"s_cselect_b32 s92, 2, 0\n\t" \
"s_sub_u32 s90, s90, s92\n\t" \
"v_mfma_f32_16x16x32_f16 a[60:63], v[80:83], v[228:231], a[60:63]\n\t" \
"s_max_i32 s90, s90, 322\n\t" \
"s_add_u32 s57, s56, 0x800000\n\t" \
"v_mfma_f32_16x16x32_f16 a[60:63], v[112:115], v[232:235], a[60:63]\n\t" \
"buffer_load_dwordx4 v[172:175], %[woff], s[36:39], s57 offen nt\n\t" \
"v_mfma_f32_16x16x32_f16 a[56:59], v[112:115], v[224:227], a[56:59]\n\t" \
"ds_read_b128 v[208:211], v248 offset:8192\n\t" \
"v_mfma_f32_16x16x32_f16 a[56:59], v[80:83], v[220:223], a[56:59]\n\t" \
"v_mfma_f32_16x16x32_f16 a[88:91], v[84:87], v[220:223], a[88:91]\n\t" \
"v_mfma_f32_16x16x32_f16 a[88:91], v[116:119], v[224:227], a[88:91]\n\t" \
"v_mfma_f32_16x16x32_f16 a[92:95], v[116:119], v[232:235], a[92:95]\n\t" \
"ds_read_b128 v[212:215], v248 offset:1024\n\t" \
"v_mfma_f32_16x16x32_f16 a[92:95], v[84:87], v[228:231], a[92:95]\n\t" \
"s_add_u32 s58, s56, 0x820000\n\t" \
"v_mfma_f32_16x16x32_f16 a[124:127], v[88:91], v[228:231], a[124:127]\n\t" \
"buffer_load_dwordx4 v[176:179], %[woff], s[36:39], s58 offen nt\n\t" \
"v_mfma_f32_16x16x32_f16 a[124:127], v[120:123], v[232:235], a[124:127]\n\t" \
"s_cmp_le_i32 s91, 7\n\t" \
"s_cbranch_scc1 PC7_%=\n\t" \
"s_sleep 10\n\t" \
"PC7_%=:\n\t" \
"v_mfma_f32_16x16x32_f16 a[120:123], v[120:123], v[224:227], a[120:123]\n\t" \
"ds_read_b128 v[216:219], v248 offset:9216\n\t" \
"v_mfma_f32_16x16x32_f16 a[120:123], v[88:91], v[220:223], a[120:123]\n\t" \
"v_mfma_f32_16x16x32_f16 a[152:155], v[92:95], v[220:223], a[152:155]\n\t" \
"v_mfma_f32_16x16x32_f16 a[152:155], v[124:127], v[224:227], a[152:155]\n\t" \
"v_mfma_f32_16x16x32_f16 a[156:159], v[124:127], v[232:235], a[156:159]\n\t" \
"s_add_u32 s57, s56, 0x840000\n\t" \
"v_mfma_f32_16x16x32_f16 a[156:159], v[92:95], v[228:231], a[156:159]\n\t" \
"buffer_load_dwordx4 v[180:183], %[woff], s[36:39], s57 offen nt\n\t" \
"v_mfma_f32_16x16x32_f16 a[188:191], v[96:99], v[228:231], a[188:191]\n\t" \
"v_mfma_f32_16x16x32_f16 a[188:191], v[128:131], v[232:235], a[188:191]\n\t" \
"v_mfma_f32_16x16x32_f16 a[184:187], v[128:131], v[224:227], a[184:187]\n\t" \
"v_mfma_f32_16x16x32_f16 a[184:187], v[96:99], v[220:223], a[184:187]\n\t" \
"v_mfma_f32_16x16x32_f16 a[216:219], v[100:103], v[220:223], a[216:219]\n\t" \
"v_mfma_f32_16x16x32_f16 a[216:219], v[132:135], v[224:227], a[216:219]\n\t" \
"s_add_u32 s58, s56, 0x860000\n\t" \
"v_mfma_f32_16x16x32_f16 a[220:223], v[132:135], v[232:235], a[220:223]\n\t" \
"buffer_load_dwordx4 v[184:187], %[woff], s[36:39], s58 offen nt\n\t" \
"v_mfma_f32_16x16x32_f16 a[220:223], v[100:103], v[228:231], a[220:223]\n\t" \
"v_mfma_f32_16x16x32_f16 a[252:255], v[104:107], v[228:231], a[252:255]\n\t" \
"v_mfma_f32_16x16x32_f16 a[252:255], v[136:139], v[232:235], a[252:255]\n\t" \
"v_mfma_f32_16x16x32_f16 a[248:251], v[136:139], v[224:227], a[248:251]\n\t" \
"v_mfma_f32_16x16x32_f16 a[248:251], v[104:107], v[220:223], a[248:251]\n\t" \
"s_sub_u32 s62, s62, 1\n\t" \
"s_cmp_lg_u32 s62, 1\n\t" \
"s_cbranch_scc1 GLOOP_%=\n\t" \
"s_cmp_lg_u32 s69, 0\n\t" \
"s_cbranch_scc1 NANOUT_%=\n\t" \
"s_waitcnt vmcnt(0)\n\t" \
"s_waitcnt lgkmcnt(0)\n\t" \
"s_waitcnt vmcnt(25)\n\t" \
"v_mfma_f32_16x16x32_f16 a[0:3], v[12:15], v[204:207], a[0:3]\n\t" \
"v_mfma_f32_16x16x32_f16 a[0:3], v[44:47], v[208:211], a[0:3]\n\t" \
"buffer_load_dwordx4 v[76:79], %[xoff], s[40:43], s59 offen offset:0 \n\t" \
"v_mfma_f32_16x16x32_f16 a[4:7], v[44:47], v[216:219], a[4:7]\n\t" \
"ds_read_b128 v[220:223], v248 offset:2048\n\t" \
"v_mfma_f32_16x16x32_f16 a[4:7], v[12:15], v[212:215], a[4:7]\n\t" \
"v_mfma_f32_16x16x32_f16 a[36:39], v[16:19], v[212:215], a[36:39]\n\t" \
"buffer_load_dwordx4 v[108:111], %[xoff], s[40:43], s59 offen offset:1024 \n\t" \
"v_mfma_f32_16x16x32_f16 a[36:39], v[48:51], v[216:219], a[36:39]\n\t" \
"v_mfma_f32_16x16x32_f16 a[32:35], v[48:51], v[208:211], a[32:35]\n\t" \
"ds_read_b128 v[224:227], v248 offset:10240\n\t" \
"v_mfma_f32_16x16x32_f16 a[32:35], v[16:19], v[204:207], a[32:35]\n\t" \
"buffer_load_dwordx4 v[80:83], %[xoff], s[40:43], s59 offen offset:2048 \n\t" \
"s_waitcnt vmcnt(24)\n\t" \
"v_mfma_f32_16x16x32_f16 a[64:67], v[20:23], v[204:207], a[64:67]\n\t" \
"v_mfma_f32_16x16x32_f16 a[64:67], v[52:55], v[208:211], a[64:67]\n\t" \
"v_mfma_f32_16x16x32_f16 a[68:71], v[52:55], v[216:219], a[68:71]\n\t" \
"buffer_load_dwordx4 v[112:115], %[xoff], s[40:43], s59 offen offset:3072 \n\t" \
"ds_read_b128 v[228:231], v248 offset:3072\n\t" \
"v_mfma_f32_16x16x32_f16 a[68:71], v[20:23], v[212:215], a[68:71]\n\t" \
"s_add_u32 s59, s59, 4096\n\t" \
"v_mfma_f32_16x16x32_f16 a[100:103], v[24:27], v[212:215], a[100:103]\n\t" \
"v_mfma_f32_16x16x32_f16 a[100:103], v[56:59], v[216:219], a[100:103]\n\t" \
"buffer_load_dwordx4 v[84:87], %[xoff], s[40:43], s59 offen offset:0 \n\t" \
"v_mfma_f32_16x16x32_f16 a[96:99], v[56:59], v[208:211], a[96:99]\n\t" \
"ds_read_b128 v[232:235], v248 offset:11264\n\t" \
"v_mfma_f32_16x16x32_f16 a[96:99], v[24:27], v[204:207], a[96:99]\n\t" \
"s_waitcnt vmcnt(22)\n\t" \
"v_mfma_f32_16x16x32_f16 a[128:131], v[28:31], v[204:207], a[128:131]\n\t" \
"buffer_load_dwordx4 v[116:119], %[xoff], s[40:43], s59 offen offset:1024 \n\t" \
"v_mfma_f32_16x16x32_f16 a[128:131], v[60:63], v[208:211], a[128:131]\n\t" \
"v_mfma_f32_16x16x32_f16 a[132:135], v[60:63], v[216:219], a[132:135]\n\t" \
"v_mfma_f32_16x16x32_f16 a[132:135], v[28:31], v[212:215], a[132:135]\n\t" \
"buffer_load_dwordx4 v[88:91], %[xoff], s[40:43], s59 offen offset:2048 \n\t" \
"v_mfma_f32_16x16x32_f16 a[164:167], v[32:35], v[212:215], a[164:167]\n\t" \
"v_mfma_f32_16x16x32_f16 a[164:167], v[64:67], v[216:219], a[164:167]\n\t" \
"v_mfma_f32_16x16x32_f16 a[160:163], v[64:67], v[208:211], a[160:163]\n\t" \
"buffer_load_dwordx4 v[120:123], %[xoff], s[40:43], s59 offen offset:3072 \n\t" \
"v_mfma_f32_16x16x32_f16 a[160:163], v[32:35], v[204:207], a[160:163]\n\t" \
"s_add_u32 s59, s59, 4096\n\t" \
"s_waitcnt vmcnt(21)\n\t" \
"v_mfma_f32_16x16x32_f16 a[192:195], v[36:39], v[204:207], a[192:195]\n\t" \
"v_mfma_f32_16x16x32_f16 a[192:195], v[68:71], v[208:211], a[192:195]\n\t" \
"buffer_load_dwordx4 v[92:95], %[xoff], s[40:43], s59 offen offset:0 \n\t" \
"v_mfma_f32_16x16x32_f16 a[196:199], v[68:71], v[216:219], a[196:199]\n\t" \
"v_mfma_f32_16x16x32_f16 a[196:199], v[36:39], v[212:215], a[196:199]\n\t" \
"v_mfma_f32_16x16x32_f16 a[228:231], v[40:43], v[212:215], a[228:231]\n\t" \
"buffer_load_dwordx4 v[124:127], %[xoff], s[40:43], s59 offen offset:1024 \n\t" \
"v_mfma_f32_16x16x32_f16 a[228:231], v[72:75], v[216:219], a[228:231]\n\t" \
"v_mfma_f32_16x16x32_f16 a[224:227], v[72:75], v[208:211], a[224:227]\n\t" \
"v_mfma_f32_16x16x32_f16 a[224:227], v[40:43], v[204:207], a[224:227]\n\t" \
"buffer_load_dwordx4 v[96:99], %[xoff], s[40:43], s59 offen offset:2048 \n\t" \
"s_waitcnt lgkmcnt(0)\n\t" \
"v_mfma_f32_16x16x32_f16 a[8:11], v[12:15], v[220:223], a[8:11]\n\t" \
"v_mfma_f32_16x16x32_f16 a[8:11], v[44:47], v[224:227], a[8:11]\n\t" \
"v_mfma_f32_16x16x32_f16 a[12:15], v[44:47], v[232:235], a[12:15]\n\t" \
"buffer_load_dwordx4 v[128:131], %[xoff], s[40:43], s59 offen offset:3072 \n\t" \
"ds_read_b128 v[204:207], v248 offset:4096\n\t" \
"v_mfma_f32_16x16x32_f16 a[12:15], v[12:15], v[228:231], a[12:15]\n\t" \
"s_add_u32 s59, s59, 4096\n\t" \
"v_mfma_f32_16x16x32_f16 a[44:47], v[16:19], v[228:231], a[44:47]\n\t" \
"v_mfma_f32_16x16x32_f16 a[44:47], v[48:51], v[232:235], a[44:47]\n\t" \
"buffer_load_dwordx4 v[100:103], %[xoff], s[40:43], s59 offen offset:0 \n\t" \
"v_mfma_f32_16x16x32_f16 a[40:43], v[48:51], v[224:227], a[40:43]\n\t" \
"ds_read_b128 v[208:211], v248 offset:12288\n\t" \
"v_mfma_f32_16x16x32_f16 a[40:43], v[16:19], v[220:223], a[40:43]\n\t" \
"v_mfma_f32_16x16x32_f16 a[72:75], v[20:23], v[220:223], a[72:75]\n\t" \
"buffer_load_dwordx4 v[132:135], %[xoff], s[40:43], s59 offen offset:1024 \n\t" \
"v_mfma_f32_16x16x32_f16 a[72:75], v[52:55], v[224:227], a[72:75]\n\t" \
"v_mfma_f32_16x16x32_f16 a[76:79], v[52:55], v[232:235], a[76:79]\n\t" \
"ds_read_b128 v[212:215], v248 offset:5120\n\t" \
"v_mfma_f32_16x16x32_f16 a[76:79], v[20:23], v[228:231], a[76:79]\n\t" \
"buffer_load_dwordx4 v[104:107], %[xoff], s[40:43], s59 offen offset:2048 \n\t" \
"v_mfma_f32_16x16x32_f16 a[108:111], v[24:27], v[228:231], a[108:111]\n\t" \
"v_mfma_f32_16x16x32_f16 a[108:111], v[56:59], v[232:235], a[108:111]\n\t" \
"v_mfma_f32_16x16x32_f16 a[104:107], v[56:59], v[224:227], a[104:107]\n\t" \
"buffer_load_dwordx4 v[136:139], %[xoff], s[40:43], s59 offen offset:3072 \n\t" \
"ds_read_b128 v[216:219], v248 offset:13312\n\t" \
"v_mfma_f32_16x16x32_f16 a[104:107], v[24:27], v[220:223], a[104:107]\n\t" \
"v_mfma_f32_16x16x32_f16 a[136:139], v[28:31], v[220:223], a[136:139]\n\t" \
"v_mfma_f32_16x16x32_f16 a[136:139], v[60:63], v[224:227], a[136:139]\n\t" \
"v_mfma_f32_16x16x32_f16 a[140:143], v[60:63], v[232:235], a[140:143]\n\t" \
"v_mfma_f32_16x16x32_f16 a[140:143], v[28:31], v[228:231], a[140:143]\n\t" \
"v_mfma_f32_16x16x32_f16 a[172:175], v[32:35], v[228:231], a[172:175]\n\t" \
"ds_read_b128 v[140:143], v248 offset:16384\n\t" \
"v_mfma_f32_16x16x32_f16 a[172:175], v[64:67], v[232:235], a[172:175]\n\t" \
"v_mfma_f32_16x16x32_f16 a[168:171], v[64:67], v[224:227], a[168:171]\n\t" \
"v_mfma_f32_16x16x32_f16 a[168:171], v[32:35], v[220:223], a[168:171]\n\t" \
"v_mfma_f32_16x16x32_f16 a[200:203], v[36:39], v[220:223], a[200:203]\n\t" \
"ds_read_b128 v[144:147], v248 offset:24576\n\t" \
"v_mfma_f32_16x16x32_f16 a[200:203], v[68:71], v[224:227], a[200:203]\n\t" \
"v_mfma_f32_16x16x32_f16 a[204:207], v[68:71], v[232:235], a[204:207]\n\t" \
"v_mfma_f32_16x16x32_f16 a[204:207], v[36:39], v[228:231], a[204:207]\n\t" \
"v_mfma_f32_16x16x32_f16 a[236:239], v[40:43], v[228:231], a[236:239]\n\t" \
"ds_read_b128 v[148:151], v248 offset:17408\n\t" \
"v_mfma_f32_16x16x32_f16 a[236:239], v[72:75], v[232:235], a[236:239]\n\t" \
"v_mfma_f32_16x16x32_f16 a[232:235], v[72:75], v[224:227], a[232:235]\n\t" \
"v_mfma_f32_16x16x32_f16 a[232:235], v[40:43], v[220:223], a[232:235]\n\t" \
"s_waitcnt lgkmcnt(3)\n\t" \
"v_mfma_f32_16x16x32_f16 a[16:19], v[12:15], v[204:207], a[16:19]\n\t" \
"ds_read_b128 v[152:155], v248 offset:25600\n\t" \
"v_mfma_f32_16x16x32_f16 a[16:19], v[44:47], v[208:211], a[16:19]\n\t" \
"v_mfma_f32_16x16x32_f16 a[20:23], v[44:47], v[216:219], a[20:23]\n\t" \
"ds_read_b128 v[220:223], v248 offset:6144\n\t" \
"v_mfma_f32_16x16x32_f16 a[20:23], v[12:15], v[212:215], a[20:23]\n\t" \
"v_mfma_f32_16x16x32_f16 a[52:55], v[16:19], v[212:215], a[52:55]\n\t" \
"ds_read_b128 v[156:159], v248 offset:18432\n\t" \
"v_mfma_f32_16x16x32_f16 a[52:55], v[48:51], v[216:219], a[52:55]\n\t" \
"v_mfma_f32_16x16x32_f16 a[48:51], v[48:51], v[208:211], a[48:51]\n\t" \
"ds_read_b128 v[224:227], v248 offset:14336\n\t" \
"v_mfma_f32_16x16x32_f16 a[48:51], v[16:19], v[204:207], a[48:51]\n\t" \
"v_mfma_f32_16x16x32_f16 a[80:83], v[20:23], v[204:207], a[80:83]\n\t" \
"ds_read_b128 v[160:163], v248 offset:26624\n\t" \
"v_mfma_f32_16x16x32_f16 a[80:83], v[52:55], v[208:211], a[80:83]\n\t" \
"v_mfma_f32_16x16x32_f16 a[84:87], v[52:55], v[216:219], a[84:87]\n\t" \
"ds_read_b128 v[228:231], v248 offset:7168\n\t" \
"v_mfma_f32_16x16x32_f16 a[84:87], v[20:23], v[212:215], a[84:87]\n\t" \
"v_mfma_f32_16x16x32_f16 a[116:119], v[24:27], v[212:215], a[116:119]\n\t" \
"ds_read_b128 v[164:167], v248 offset:19456\n\t" \
"v_mfma_f32_16x16x32_f16 a[116:119], v[56:59], v[216:219], a[116:119]\n\t" \
"v_mfma_f32_16x16x32_f16 a[112:115], v[56:59], v[208:211], a[112:115]\n\t" \
"ds_read_b128 v[232:235], v248 offset:15360\n\t" \
"v_mfma_f32_16x16x32_f16 a[112:115], v[24:27], v[204:207], a[112:115]\n\t" \
"v_mfma_f32_16x16x32_f16 a[144:147], v[28:31], v[204:207], a[144:147]\n\t" \
"ds_read_b128 v[168:171], v248 offset:27648\n\t" \
"v_mfma_f32_16x16x32_f16 a[144:147], v[60:63], v[208:211], a[144:147]\n\t" \
"v_mfma_f32_16x16x32_f16 a[148:151], v[60:63], v[216:219], a[148:151]\n\t" \
"v_mfma_f32_16x16x32_f16 a[148:151], v[28:31], v[212:215], a[148:151]\n\t" \
"v_mfma_f32_16x16x32_f16 a[180:183], v[32:35], v[212:215], a[180:183]\n\t" \
"ds_read_b128 v[172:175], v248 offset:20480\n\t" \
"v_mfma_f32_16x16x32_f16 a[180:183], v[64:67], v[216:219], a[180:183]\n\t" \
"v_mfma_f32_16x16x32_f16 a[176:179], v[64:67], v[208:211], a[176:179]\n\t" \
"v_mfma_f32_16x16x32_f16 a[176:179], v[32:35], v[204:207], a[176:179]\n\t" \
"v_mfma_f32_16x16x32_f16 a[208:211], v[36:39], v[204:207], a[208:211]\n\t" \
"ds_read_b128 v[176:179], v248 offset:28672\n\t" \
"v_mfma_f32_16x16x32_f16 a[208:211], v[68:71], v[208:211], a[208:211]\n\t" \
"v_mfma_f32_16x16x32_f16 a[212:215], v[68:71], v[216:219], a[212:215]\n\t" \
"v_mfma_f32_16x16x32_f16 a[212:215], v[36:39], v[212:215], a[212:215]\n\t" \
"v_mfma_f32_16x16x32_f16 a[244:247], v[40:43], v[212:215], a[244:247]\n\t" \
"ds_read_b128 v[180:183], v248 offset:21504\n\t" \
"v_mfma_f32_16x16x32_f16 a[244:247], v[72:75], v[216:219], a[244:247]\n\t" \
"v_mfma_f32_16x16x32_f16 a[240:243], v[72:75], v[208:211], a[240:243]\n\t" \
"v_mfma_f32_16x16x32_f16 a[240:243], v[40:43], v[204:207], a[240:243]\n\t" \
"s_waitcnt lgkmcnt(4)\n\t" \
"v_mfma_f32_16x16x32_f16 a[24:27], v[12:15], v[220:223], a[24:27]\n\t" \
"ds_read_b128 v[184:187], v248 offset:29696\n\t" \
"v_mfma_f32_16x16x32_f16 a[24:27], v[44:47], v[224:227], a[24:27]\n\t" \
"v_mfma_f32_16x16x32_f16 a[28:31], v[44:47], v[232:235], a[28:31]\n\t" \
"v_mfma_f32_16x16x32_f16 a[28:31], v[12:15], v[228:231], a[28:31]\n\t" \
"v_mfma_f32_16x16x32_f16 a[60:63], v[16:19], v[228:231], a[60:63]\n\t" \
"ds_read_b128 v[188:191], v248 offset:22528\n\t" \
"v_mfma_f32_16x16x32_f16 a[60:63], v[48:51], v[232:235], a[60:63]\n\t" \
"v_mfma_f32_16x16x32_f16 a[56:59], v[48:51], v[224:227], a[56:59]\n\t" \
"v_mfma_f32_16x16x32_f16 a[56:59], v[16:19], v[220:223], a[56:59]\n\t" \
"v_mfma_f32_16x16x32_f16 a[88:91], v[20:23], v[220:223], a[88:91]\n\t" \
"ds_read_b128 v[192:195], v248 offset:30720\n\t" \
"v_mfma_f32_16x16x32_f16 a[88:91], v[52:55], v[224:227], a[88:91]\n\t" \
"v_mfma_f32_16x16x32_f16 a[92:95], v[52:55], v[232:235], a[92:95]\n\t" \
"v_mfma_f32_16x16x32_f16 a[92:95], v[20:23], v[228:231], a[92:95]\n\t" \
"v_mfma_f32_16x16x32_f16 a[124:127], v[24:27], v[228:231], a[124:127]\n\t" \
"ds_read_b128 v[196:199], v248 offset:23552\n\t" \
"v_mfma_f32_16x16x32_f16 a[124:127], v[56:59], v[232:235], a[124:127]\n\t" \
"v_mfma_f32_16x16x32_f16 a[120:123], v[56:59], v[224:227], a[120:123]\n\t" \
"v_mfma_f32_16x16x32_f16 a[120:123], v[24:27], v[220:223], a[120:123]\n\t" \
"v_mfma_f32_16x16x32_f16 a[152:155], v[28:31], v[220:223], a[152:155]\n\t" \
"ds_read_b128 v[200:203], v248 offset:31744\n\t" \
"v_mfma_f32_16x16x32_f16 a[152:155], v[60:63], v[224:227], a[152:155]\n\t" \
"v_mfma_f32_16x16x32_f16 a[156:159], v[60:63], v[232:235], a[156:159]\n\t" \
"v_mfma_f32_16x16x32_f16 a[156:159], v[28:31], v[228:231], a[156:159]\n\t" \
"v_mfma_f32_16x16x32_f16 a[188:191], v[32:35], v[228:231], a[188:191]\n\t" \
"v_mfma_f32_16x16x32_f16 a[188:191], v[64:67], v[232:235], a[188:191]\n\t" \
"v_mfma_f32_16x16x32_f16 a[184:187], v[64:67], v[224:227], a[184:187]\n\t" \
"v_mfma_f32_16x16x32_f16 a[184:187], v[32:35], v[220:223], a[184:187]\n\t" \
"v_mfma_f32_16x16x32_f16 a[216:219], v[36:39], v[220:223], a[216:219]\n\t" \
"v_mfma_f32_16x16x32_f16 a[216:219], v[68:71], v[224:227], a[216:219]\n\t" \
"v_mfma_f32_16x16x32_f16 a[220:223], v[68:71], v[232:235], a[220:223]\n\t" \
"v_mfma_f32_16x16x32_f16 a[220:223], v[36:39], v[228:231], a[220:223]\n\t" \
"v_mfma_f32_16x16x32_f16 a[252:255], v[40:43], v[228:231], a[252:255]\n\t" \
"v_mfma_f32_16x16x32_f16 a[252:255], v[72:75], v[232:235], a[252:255]\n\t" \
"v_mfma_f32_16x16x32_f16 a[248:251], v[72:75], v[224:227], a[248:251]\n\t" \
"v_mfma_f32_16x16x32_f16 a[248:251], v[40:43], v[220:223], a[248:251]\n\t" \
"s_waitcnt vmcnt(0)\n\t" \
"s_waitcnt lgkmcnt(8)\n\t" \
"v_mfma_f32_16x16x32_f16 a[0:3], v[76:79], v[140:143], a[0:3]\n\t" \
"v_mfma_f32_16x16x32_f16 a[0:3], v[108:111], v[144:147], a[0:3]\n\t" \
"v_mfma_f32_16x16x32_f16 a[4:7], v[108:111], v[152:155], a[4:7]\n\t" \
"v_mfma_f32_16x16x32_f16 a[4:7], v[76:79], v[148:151], a[4:7]\n\t" \
"v_mfma_f32_16x16x32_f16 a[8:11], v[76:79], v[156:159], a[8:11]\n\t" \
"v_mfma_f32_16x16x32_f16 a[8:11], v[108:111], v[160:163], a[8:11]\n\t" \
"v_mfma_f32_16x16x32_f16 a[12:15], v[108:111], v[168:171], a[12:15]\n\t" \
"v_mfma_f32_16x16x32_f16 a[12:15], v[76:79], v[164:167], a[12:15]\n\t" \
"v_mfma_f32_16x16x32_f16 a[44:47], v[80:83], v[164:167], a[44:47]\n\t" \
"v_mfma_f32_16x16x32_f16 a[44:47], v[112:115], v[168:171], a[44:47]\n\t" \
"s_nop 5\n\t" \
"v_accvgpr_read_b32 v12, a0\n\t" \
"v_accvgpr_read_b32 v13, a4\n\t" \
"v_accvgpr_read_b32 v14, a8\n\t" \
"v_accvgpr_read_b32 v15, a12\n\t" \
"s_mov_b32 s65, 0\n\t" \
"v_max_f32 v12, 0, v12\n\t" \
"v_mfma_f32_16x16x32_f16 a[40:43], v[112:115], v[160:163], a[40:43]\n\t" \
"v_max_f32 v13, 0, v13\n\t" \
"v_max_f32 v14, 0, v14\n\t" \
"v_max_f32 v15, 0, v15\n\t" \
"buffer_store_dwordx4 v[12:15], %[ooff], s[44:47], s65 offen offset:0 \n\t" \
"v_accvgpr_read_b32 v16, a1\n\t" \
"v_accvgpr_read_b32 v17, a5\n\t" \
"v_mfma_f32_16x16x32_f16 a[40:43], v[80:83], v[156:159], a[40:43]\n\t" \
"v_accvgpr_read_b32 v18, a9\n\t" \
"v_accvgpr_read_b32 v19, a13\n\t" \
"s_mov_b32 s63, 131072\n\t" \
"v_max_f32 v16, 0, v16\n\t" \
"v_max_f32 v17, 0, v17\n\t" \
"v_max_f32 v18, 0, v18\n\t" \
"v_mfma_f32_16x16x32_f16 a[36:39], v[80:83], v[148:151], a[36:39]\n\t" \
"v_max_f32 v19, 0, v19\n\t" \
"buffer_store_dwordx4 v[16:19], %[ooff], s[44:47], s63 offen offset:0 \n\t" \
"v_accvgpr_read_b32 v20, a2\n\t" \
"v_accvgpr_read_b32 v21, a6\n\t" \
"v_accvgpr_read_b32 v22, a10\n\t" \
"v_accvgpr_read_b32 v23, a14\n\t" \
"v_mfma_f32_16x16x32_f16 a[36:39], v[112:115], v[152:155], a[36:39]\n\t" \
"s_mov_b32 s65, 262144\n\t" \
"v_max_f32 v20, 0, v20\n\t" \
"v_max_f32 v21, 0, v21\n\t" \
"v_max_f32 v22, 0, v22\n\t" \
"v_max_f32 v23, 0, v23\n\t" \
"buffer_store_dwordx4 v[20:23], %[ooff], s[44:47], s65 offen offset:0 \n\t" \
"v_mfma_f32_16x16x32_f16 a[32:35], v[112:115], v[144:147], a[32:35]\n\t" \
"v_accvgpr_read_b32 v24, a3\n\t" \
"v_accvgpr_read_b32 v25, a7\n\t" \
"v_accvgpr_read_b32 v26, a11\n\t" \
"v_accvgpr_read_b32 v27, a15\n\t" \
"s_mov_b32 s63, 393216\n\t" \
"v_max_f32 v24, 0, v24\n\t" \
"v_mfma_f32_16x16x32_f16 a[32:35], v[80:83], v[140:143], a[32:35]\n\t" \
"v_max_f32 v25, 0, v25\n\t" \
"v_max_f32 v26, 0, v26\n\t" \
"v_max_f32 v27, 0, v27\n\t" \
"buffer_store_dwordx4 v[24:27], %[ooff], s[44:47], s63 offen offset:0 \n\t" \
"v_mfma_f32_16x16x32_f16 a[64:67], v[84:87], v[140:143], a[64:67]\n\t" \
"v_mfma_f32_16x16x32_f16 a[64:67], v[116:119], v[144:147], a[64:67]\n\t" \
"s_nop 5\n\t" \
"v_accvgpr_read_b32 v28, a32\n\t" \
"v_accvgpr_read_b32 v29, a36\n\t" \
"v_accvgpr_read_b32 v30, a40\n\t" \
"v_accvgpr_read_b32 v31, a44\n\t" \
"s_mov_b32 s65, 2097152\n\t" \
"v_max_f32 v28, 0, v28\n\t" \
"v_mfma_f32_16x16x32_f16 a[68:71], v[116:119], v[152:155], a[68:71]\n\t" \
"v_max_f32 v29, 0, v29\n\t" \
"v_max_f32 v30, 0, v30\n\t" \
"v_max_f32 v31, 0, v31\n\t" \
"buffer_store_dwordx4 v[28:31], %[ooff], s[44:47], s65 offen offset:0 \n\t" \
"v_accvgpr_read_b32 v32, a33\n\t" \
"v_accvgpr_read_b32 v33, a37\n\t" \
"v_mfma_f32_16x16x32_f16 a[68:71], v[84:87], v[148:151], a[68:71]\n\t" \
"v_accvgpr_read_b32 v34, a41\n\t" \
"v_accvgpr_read_b32 v35, a45\n\t" \
"s_mov_b32 s63, 2228224\n\t" \
"v_max_f32 v32, 0, v32\n\t" \
"v_max_f32 v33, 0, v33\n\t" \
"v_max_f32 v34, 0, v34\n\t" \
"v_mfma_f32_16x16x32_f16 a[72:75], v[84:87], v[156:159], a[72:75]\n\t" \
"v_max_f32 v35, 0, v35\n\t" \
"buffer_store_dwordx4 v[32:35], %[ooff], s[44:47], s63 offen offset:0 \n\t" \
"v_accvgpr_read_b32 v36, a34\n\t" \
"v_accvgpr_read_b32 v37, a38\n\t" \
"v_accvgpr_read_b32 v38, a42\n\t" \
"v_accvgpr_read_b32 v39, a46\n\t" \
"v_mfma_f32_16x16x32_f16 a[72:75], v[116:119], v[160:163], a[72:75]\n\t" \
"s_mov_b32 s65, 2359296\n\t" \
"v_max_f32 v36, 0, v36\n\t" \
"v_max_f32 v37, 0, v37\n\t" \
"v_max_f32 v38, 0, v38\n\t" \
"v_max_f32 v39, 0, v39\n\t" \
"buffer_store_dwordx4 v[36:39], %[ooff], s[44:47], s65 offen offset:0 \n\t" \
"v_mfma_f32_16x16x32_f16 a[76:79], v[116:119], v[168:171], a[76:79]\n\t" \
"v_accvgpr_read_b32 v40, a35\n\t" \
"v_accvgpr_read_b32 v41, a39\n\t" \
"v_accvgpr_read_b32 v42, a43\n\t" \
"v_accvgpr_read_b32 v43, a47\n\t" \
"s_mov_b32 s63, 2490368\n\t" \
"v_max_f32 v40, 0, v40\n\t" \
"v_mfma_f32_16x16x32_f16 a[76:79], v[84:87], v[164:167], a[76:79]\n\t" \
"v_max_f32 v41, 0, v41\n\t" \
"v_max_f32 v42, 0, v42\n\t" \
"v_max_f32 v43, 0, v43\n\t" \
"buffer_store_dwordx4 v[40:43], %[ooff], s[44:47], s63 offen offset:0 \n\t" \
"v_mfma_f32_16x16x32_f16 a[108:111], v[88:91], v[164:167], a[108:111]\n\t" \
"v_mfma_f32_16x16x32_f16 a[108:111], v[120:123], v[168:171], a[108:111]\n\t" \
"s_nop 5\n\t" \
"v_accvgpr_read_b32 v44, a64\n\t" \
"v_accvgpr_read_b32 v45, a68\n\t" \
"v_accvgpr_read_b32 v46, a72\n\t" \
"v_accvgpr_read_b32 v47, a76\n\t" \
"s_mov_b32 s65, 4194304\n\t" \
"v_max_f32 v44, 0, v44\n\t" \
"v_mfma_f32_16x16x32_f16 a[104:107], v[120:123], v[160:163], a[104:107]\n\t" \
"v_max_f32 v45, 0, v45\n\t" \
"v_max_f32 v46, 0, v46\n\t" \
"v_max_f32 v47, 0, v47\n\t" \
"buffer_store_dwordx4 v[44:47], %[ooff], s[44:47], s65 offen offset:0 \n\t" \
"v_accvgpr_read_b32 v48, a65\n\t" \
"v_accvgpr_read_b32 v49, a69\n\t" \
"v_mfma_f32_16x16x32_f16 a[104:107], v[88:91], v[156:159], a[104:107]\n\t" \
"v_accvgpr_read_b32 v50, a73\n\t" \
"v_accvgpr_read_b32 v51, a77\n\t" \
"s_mov_b32 s63, 4325376\n\t" \
"v_max_f32 v48, 0, v48\n\t" \
"v_max_f32 v49, 0, v49\n\t" \
"v_max_f32 v50, 0, v50\n\t" \
"v_mfma_f32_16x16x32_f16 a[100:103], v[88:91], v[148:151], a[100:103]\n\t" \
"v_max_f32 v51, 0, v51\n\t" \
"buffer_store_dwordx4 v[48:51], %[ooff], s[44:47], s63 offen offset:0 \n\t" \
"v_accvgpr_read_b32 v52, a66\n\t" \
"v_accvgpr_read_b32 v53, a70\n\t" \
"v_accvgpr_read_b32 v54, a74\n\t" \
"v_accvgpr_read_b32 v55, a78\n\t" \
"v_mfma_f32_16x16x32_f16 a[100:103], v[120:123], v[152:155], a[100:103]\n\t" \
"s_mov_b32 s65, 4456448\n\t" \
"v_max_f32 v52, 0, v52\n\t" \
"v_max_f32 v53, 0, v53\n\t" \
"v_max_f32 v54, 0, v54\n\t" \
"v_max_f32 v55, 0, v55\n\t" \
"buffer_store_dwordx4 v[52:55], %[ooff], s[44:47], s65 offen offset:0 \n\t" \
"v_mfma_f32_16x16x32_f16 a[96:99], v[120:123], v[144:147], a[96:99]\n\t" \
"v_accvgpr_read_b32 v56, a67\n\t" \
"v_accvgpr_read_b32 v57, a71\n\t" \
"v_accvgpr_read_b32 v58, a75\n\t" \
"v_accvgpr_read_b32 v59, a79\n\t" \
"s_mov_b32 s63, 4587520\n\t" \
"v_max_f32 v56, 0, v56\n\t" \
"v_mfma_f32_16x16x32_f16 a[96:99], v[88:91], v[140:143], a[96:99]\n\t" \
"v_max_f32 v57, 0, v57\n\t" \
"v_max_f32 v58, 0, v58\n\t" \
"v_max_f32 v59, 0, v59\n\t" \
"buffer_store_dwordx4 v[56:59], %[ooff], s[44:47], s63 offen offset:0 \n\t" \
"v_mfma_f32_16x16x32_f16 a[128:131], v[92:95], v[140:143], a[128:131]\n\t" \
"v_mfma_f32_16x16x32_f16 a[128:131], v[124:127], v[144:147], a[128:131]\n\t" \
"s_nop 5\n\t" \
"v_accvgpr_read_b32 v60, a96\n\t" \
"v_accvgpr_read_b32 v61, a100\n\t" \
"v_accvgpr_read_b32 v62, a104\n\t" \
"v_accvgpr_read_b32 v63, a108\n\t" \
"s_mov_b32 s65, 6291456\n\t" \
"v_max_f32 v60, 0, v60\n\t" \
"v_mfma_f32_16x16x32_f16 a[132:135], v[124:127], v[152:155], a[132:135]\n\t" \
"v_max_f32 v61, 0, v61\n\t" \
"v_max_f32 v62, 0, v62\n\t" \
"v_max_f32 v63, 0, v63\n\t" \
"buffer_store_dwordx4 v[60:63], %[ooff], s[44:47], s65 offen offset:0 \n\t" \
"v_accvgpr_read_b32 v64, a97\n\t" \
"v_accvgpr_read_b32 v65, a101\n\t" \
"v_mfma_f32_16x16x32_f16 a[132:135], v[92:95], v[148:151], a[132:135]\n\t" \
"v_accvgpr_read_b32 v66, a105\n\t" \
"v_accvgpr_read_b32 v67, a109\n\t" \
"s_mov_b32 s63, 6422528\n\t" \
"v_max_f32 v64, 0, v64\n\t" \
"v_max_f32 v65, 0, v65\n\t" \
"v_max_f32 v66, 0, v66\n\t" \
"v_mfma_f32_16x16x32_f16 a[136:139], v[92:95], v[156:159], a[136:139]\n\t" \
"v_max_f32 v67, 0, v67\n\t" \
"buffer_store_dwordx4 v[64:67], %[ooff], s[44:47], s63 offen offset:0 \n\t" \
"v_accvgpr_read_b32 v68, a98\n\t" \
"v_accvgpr_read_b32 v69, a102\n\t" \
"v_accvgpr_read_b32 v70, a106\n\t" \
"v_accvgpr_read_b32 v71, a110\n\t" \
"v_mfma_f32_16x16x32_f16 a[136:139], v[124:127], v[160:163], a[136:139]\n\t" \
"s_mov_b32 s65, 6553600\n\t" \
"v_max_f32 v68, 0, v68\n\t" \
"v_max_f32 v69, 0, v69\n\t" \
"v_max_f32 v70, 0, v70\n\t" \
"v_max_f32 v71, 0, v71\n\t" \
"buffer_store_dwordx4 v[68:71], %[ooff], s[44:47], s65 offen offset:0 \n\t" \
"v_mfma_f32_16x16x32_f16 a[140:143], v[124:127], v[168:171], a[140:143]\n\t" \
"v_accvgpr_read_b32 v72, a99\n\t" \
"v_accvgpr_read_b32 v73, a103\n\t" \
"v_accvgpr_read_b32 v74, a107\n\t" \
"v_accvgpr_read_b32 v75, a111\n\t" \
"s_mov_b32 s63, 6684672\n\t" \
"v_max_f32 v72, 0, v72\n\t" \
"v_mfma_f32_16x16x32_f16 a[140:143], v[92:95], v[164:167], a[140:143]\n\t" \
"v_max_f32 v73, 0, v73\n\t" \
"v_max_f32 v74, 0, v74\n\t" \
"v_max_f32 v75, 0, v75\n\t" \
"buffer_store_dwordx4 v[72:75], %[ooff], s[44:47], s63 offen offset:0 \n\t" \
"v_mfma_f32_16x16x32_f16 a[172:175], v[96:99], v[164:167], a[172:175]\n\t" \
"v_mfma_f32_16x16x32_f16 a[172:175], v[128:131], v[168:171], a[172:175]\n\t" \
"s_nop 5\n\t" \
"v_accvgpr_read_b32 v12, a128\n\t" \
"v_accvgpr_read_b32 v13, a132\n\t" \
"v_accvgpr_read_b32 v14, a136\n\t" \
"v_accvgpr_read_b32 v15, a140\n\t" \
"s_mov_b32 s65, 8388608\n\t" \
"v_max_f32 v12, 0, v12\n\t" \
"v_mfma_f32_16x16x32_f16 a[168:171], v[128:131], v[160:163], a[168:171]\n\t" \
"v_max_f32 v13, 0, v13\n\t" \
"v_max_f32 v14, 0, v14\n\t" \
"v_max_f32 v15, 0, v15\n\t" \
"buffer_store_dwordx4 v[12:15], %[ooff], s[44:47], s65 offen offset:0 \n\t" \
"v_accvgpr_read_b32 v16, a129\n\t" \
"v_accvgpr_read_b32 v17, a133\n\t" \
"v_mfma_f32_16x16x32_f16 a[168:171], v[96:99], v[156:159], a[168:171]\n\t" \
"v_accvgpr_read_b32 v18, a137\n\t" \
"v_accvgpr_read_b32 v19, a141\n\t" \
"s_mov_b32 s63, 8519680\n\t" \
"v_max_f32 v16, 0, v16\n\t" \
"v_max_f32 v17, 0, v17\n\t" \
"v_max_f32 v18, 0, v18\n\t" \
"v_mfma_f32_16x16x32_f16 a[164:167], v[96:99], v[148:151], a[164:167]\n\t" \
"v_max_f32 v19, 0, v19\n\t" \
"buffer_store_dwordx4 v[16:19], %[ooff], s[44:47], s63 offen offset:0 \n\t" \
"v_accvgpr_read_b32 v20, a130\n\t" \
"v_accvgpr_read_b32 v21, a134\n\t" \
"v_accvgpr_read_b32 v22, a138\n\t" \
"v_accvgpr_read_b32 v23, a142\n\t" \
"v_mfma_f32_16x16x32_f16 a[164:167], v[128:131], v[152:155], a[164:167]\n\t" \
"s_mov_b32 s65, 8650752\n\t" \
"v_max_f32 v20, 0, v20\n\t" \
"v_max_f32 v21, 0, v21\n\t" \
"v_max_f32 v22, 0, v22\n\t" \
"v_max_f32 v23, 0, v23\n\t" \
"buffer_store_dwordx4 v[20:23], %[ooff], s[44:47], s65 offen offset:0 \n\t" \
"v_mfma_f32_16x16x32_f16 a[160:163], v[128:131], v[144:147], a[160:163]\n\t" \
"v_accvgpr_read_b32 v24, a131\n\t" \
"v_accvgpr_read_b32 v25, a135\n\t" \
"v_accvgpr_read_b32 v26, a139\n\t" \
"v_accvgpr_read_b32 v27, a143\n\t" \
"s_mov_b32 s63, 8781824\n\t" \
"v_max_f32 v24, 0, v24\n\t" \
"v_mfma_f32_16x16x32_f16 a[160:163], v[96:99], v[140:143], a[160:163]\n\t" \
"v_max_f32 v25, 0, v25\n\t" \
"v_max_f32 v26, 0, v26\n\t" \
"v_max_f32 v27, 0, v27\n\t" \
"buffer_store_dwordx4 v[24:27], %[ooff], s[44:47], s63 offen offset:0 \n\t" \
"v_mfma_f32_16x16x32_f16 a[192:195], v[100:103], v[140:143], a[192:195]\n\t" \
"v_mfma_f32_16x16x32_f16 a[192:195], v[132:135], v[144:147], a[192:195]\n\t" \
"s_nop 5\n\t" \
"v_accvgpr_read_b32 v28, a160\n\t" \
"v_accvgpr_read_b32 v29, a164\n\t" \
"v_accvgpr_read_b32 v30, a168\n\t" \
"v_accvgpr_read_b32 v31, a172\n\t" \
"s_mov_b32 s65, 10485760\n\t" \
"v_max_f32 v28, 0, v28\n\t" \
"v_mfma_f32_16x16x32_f16 a[196:199], v[132:135], v[152:155], a[196:199]\n\t" \
"v_max_f32 v29, 0, v29\n\t" \
"v_max_f32 v30, 0, v30\n\t" \
"v_max_f32 v31, 0, v31\n\t" \
"buffer_store_dwordx4 v[28:31], %[ooff], s[44:47], s65 offen offset:0 \n\t" \
"v_accvgpr_read_b32 v32, a161\n\t" \
"v_accvgpr_read_b32 v33, a165\n\t" \
"v_mfma_f32_16x16x32_f16 a[196:199], v[100:103], v[148:151], a[196:199]\n\t" \
"v_accvgpr_read_b32 v34, a169\n\t" \
"v_accvgpr_read_b32 v35, a173\n\t" \
"s_mov_b32 s63, 10616832\n\t" \
"v_max_f32 v32, 0, v32\n\t" \
"v_max_f32 v33, 0, v33\n\t" \
"v_max_f32 v34, 0, v34\n\t" \
"v_mfma_f32_16x16x32_f16 a[200:203], v[100:103], v[156:159], a[200:203]\n\t" \
"v_max_f32 v35, 0, v35\n\t" \
"buffer_store_dwordx4 v[32:35], %[ooff], s[44:47], s63 offen offset:0 \n\t" \
"v_accvgpr_read_b32 v36, a162\n\t" \
"v_accvgpr_read_b32 v37, a166\n\t" \
"v_accvgpr_read_b32 v38, a170\n\t" \
"v_accvgpr_read_b32 v39, a174\n\t" \
"v_mfma_f32_16x16x32_f16 a[200:203], v[132:135], v[160:163], a[200:203]\n\t" \
"s_mov_b32 s65, 10747904\n\t" \
"v_max_f32 v36, 0, v36\n\t" \
"v_max_f32 v37, 0, v37\n\t" \
"v_max_f32 v38, 0, v38\n\t" \
"v_max_f32 v39, 0, v39\n\t" \
"buffer_store_dwordx4 v[36:39], %[ooff], s[44:47], s65 offen offset:0 \n\t" \
"v_mfma_f32_16x16x32_f16 a[204:207], v[132:135], v[168:171], a[204:207]\n\t" \
"v_accvgpr_read_b32 v40, a163\n\t" \
"v_accvgpr_read_b32 v41, a167\n\t" \
"v_accvgpr_read_b32 v42, a171\n\t" \
"v_accvgpr_read_b32 v43, a175\n\t" \
"s_mov_b32 s63, 10878976\n\t" \
"v_max_f32 v40, 0, v40\n\t" \
"v_mfma_f32_16x16x32_f16 a[204:207], v[100:103], v[164:167], a[204:207]\n\t" \
"v_max_f32 v41, 0, v41\n\t" \
"v_max_f32 v42, 0, v42\n\t" \
"v_max_f32 v43, 0, v43\n\t" \
"buffer_store_dwordx4 v[40:43], %[ooff], s[44:47], s63 offen offset:0 \n\t" \
"v_mfma_f32_16x16x32_f16 a[236:239], v[104:107], v[164:167], a[236:239]\n\t" \
"v_mfma_f32_16x16x32_f16 a[236:239], v[136:139], v[168:171], a[236:239]\n\t" \
"s_nop 5\n\t" \
"v_accvgpr_read_b32 v44, a192\n\t" \
"v_accvgpr_read_b32 v45, a196\n\t" \
"v_accvgpr_read_b32 v46, a200\n\t" \
"v_accvgpr_read_b32 v47, a204\n\t" \
"s_mov_b32 s65, 12582912\n\t" \
"v_max_f32 v44, 0, v44\n\t" \
"v_mfma_f32_16x16x32_f16 a[232:235], v[136:139], v[160:163], a[232:235]\n\t" \
"v_max_f32 v45, 0, v45\n\t" \
"v_max_f32 v46, 0, v46\n\t" \
"v_max_f32 v47, 0, v47\n\t" \
"buffer_store_dwordx4 v[44:47], %[ooff], s[44:47], s65 offen offset:0 \n\t" \
"v_accvgpr_read_b32 v48, a193\n\t" \
"v_accvgpr_read_b32 v49, a197\n\t" \
"v_mfma_f32_16x16x32_f16 a[232:235], v[104:107], v[156:159], a[232:235]\n\t" \
"v_accvgpr_read_b32 v50, a201\n\t" \
"v_accvgpr_read_b32 v51, a205\n\t" \
"s_mov_b32 s63, 12713984\n\t" \
"v_max_f32 v48, 0, v48\n\t" \
"v_max_f32 v49, 0, v49\n\t" \
"v_max_f32 v50, 0, v50\n\t" \
"v_mfma_f32_16x16x32_f16 a[228:231], v[104:107], v[148:151], a[228:231]\n\t" \
"v_max_f32 v51, 0, v51\n\t" \
"buffer_store_dwordx4 v[48:51], %[ooff], s[44:47], s63 offen offset:0 \n\t" \
"v_accvgpr_read_b32 v52, a194\n\t" \
"v_accvgpr_read_b32 v53, a198\n\t" \
"v_accvgpr_read_b32 v54, a202\n\t" \
"v_accvgpr_read_b32 v55, a206\n\t" \
"v_mfma_f32_16x16x32_f16 a[228:231], v[136:139], v[152:155], a[228:231]\n\t" \
"s_mov_b32 s65, 12845056\n\t" \
"v_max_f32 v52, 0, v52\n\t" \
"v_max_f32 v53, 0, v53\n\t" \
"v_max_f32 v54, 0, v54\n\t" \
"v_max_f32 v55, 0, v55\n\t" \
"buffer_store_dwordx4 v[52:55], %[ooff], s[44:47], s65 offen offset:0 \n\t" \
"v_mfma_f32_16x16x32_f16 a[224:227], v[136:139], v[144:147], a[224:227]\n\t" \
"v_accvgpr_read_b32 v56, a195\n\t" \
"v_accvgpr_read_b32 v57, a199\n\t" \
"v_accvgpr_read_b32 v58, a203\n\t" \
"v_accvgpr_read_b32 v59, a207\n\t" \
"s_mov_b32 s63, 12976128\n\t" \
"v_max_f32 v56, 0, v56\n\t" \
"v_mfma_f32_16x16x32_f16 a[224:227], v[104:107], v[140:143], a[224:227]\n\t" \
"v_max_f32 v57, 0, v57\n\t" \
"v_max_f32 v58, 0, v58\n\t" \
"v_max_f32 v59, 0, v59\n\t" \
"buffer_store_dwordx4 v[56:59], %[ooff], s[44:47], s63 offen offset:0 \n\t" \
"s_waitcnt lgkmcnt(0)\n\t" \
"v_mfma_f32_16x16x32_f16 a[16:19], v[76:79], v[172:175], a[16:19]\n\t" \
"v_mfma_f32_16x16x32_f16 a[16:19], v[108:111], v[176:179], a[16:19]\n\t" \
"s_nop 5\n\t" \
"v_accvgpr_read_b32 v60, a224\n\t" \
"v_accvgpr_read_b32 v61, a228\n\t" \
"v_accvgpr_read_b32 v62, a232\n\t" \
"v_accvgpr_read_b32 v63, a236\n\t" \
"s_mov_b32 s65, 14680064\n\t" \
"v_max_f32 v60, 0, v60\n\t" \
"v_mfma_f32_16x16x32_f16 a[20:23], v[108:111], v[184:187], a[20:23]\n\t" \
"v_max_f32 v61, 0, v61\n\t" \
"v_max_f32 v62, 0, v62\n\t" \
"v_max_f32 v63, 0, v63\n\t" \
"buffer_store_dwordx4 v[60:63], %[ooff], s[44:47], s65 offen offset:0 \n\t" \
"v_accvgpr_read_b32 v64, a225\n\t" \
"v_accvgpr_read_b32 v65, a229\n\t" \
"v_mfma_f32_16x16x32_f16 a[20:23], v[76:79], v[180:183], a[20:23]\n\t" \
"v_accvgpr_read_b32 v66, a233\n\t" \
"v_accvgpr_read_b32 v67, a237\n\t" \
"s_mov_b32 s63, 14811136\n\t" \
"v_max_f32 v64, 0, v64\n\t" \
"v_max_f32 v65, 0, v65\n\t" \
"v_max_f32 v66, 0, v66\n\t" \
"v_mfma_f32_16x16x32_f16 a[24:27], v[76:79], v[188:191], a[24:27]\n\t" \
"v_max_f32 v67, 0, v67\n\t" \
"buffer_store_dwordx4 v[64:67], %[ooff], s[44:47], s63 offen offset:0 \n\t" \
"v_accvgpr_read_b32 v68, a226\n\t" \
"v_accvgpr_read_b32 v69, a230\n\t" \
"v_accvgpr_read_b32 v70, a234\n\t" \
"v_accvgpr_read_b32 v71, a238\n\t" \
"v_mfma_f32_16x16x32_f16 a[24:27], v[108:111], v[192:195], a[24:27]\n\t" \
"s_mov_b32 s65, 14942208\n\t" \
"v_max_f32 v68, 0, v68\n\t" \
"v_max_f32 v69, 0, v69\n\t" \
"v_max_f32 v70, 0, v70\n\t" \
"v_max_f32 v71, 0, v71\n\t" \
"buffer_store_dwordx4 v[68:71], %[ooff], s[44:47], s65 offen offset:0 \n\t" \
"v_mfma_f32_16x16x32_f16 a[28:31], v[108:111], v[200:203], a[28:31]\n\t" \
"v_accvgpr_read_b32 v72, a227\n\t" \
"v_accvgpr_read_b32 v73, a231\n\t" \
"v_accvgpr_read_b32 v74, a235\n\t" \
"v_accvgpr_read_b32 v75, a239\n\t" \
"s_mov_b32 s63, 15073280\n\t" \
"v_max_f32 v72, 0, v72\n\t" \
"v_mfma_f32_16x16x32_f16 a[28:31], v[76:79], v[196:199], a[28:31]\n\t" \
"v_max_f32 v73, 0, v73\n\t" \
"v_max_f32 v74, 0, v74\n\t" \
"v_max_f32 v75, 0, v75\n\t" \
"buffer_store_dwordx4 v[72:75], %[ooff], s[44:47], s63 offen offset:0 \n\t" \
"v_mfma_f32_16x16x32_f16 a[60:63], v[80:83], v[196:199], a[60:63]\n\t" \
"v_mfma_f32_16x16x32_f16 a[60:63], v[112:115], v[200:203], a[60:63]\n\t" \
"s_nop 5\n\t" \
"v_accvgpr_read_b32 v12, a16\n\t" \
"v_accvgpr_read_b32 v13, a20\n\t" \
"v_accvgpr_read_b32 v14, a24\n\t" \
"v_accvgpr_read_b32 v15, a28\n\t" \
"s_mov_b32 s65, 0\n\t" \
"v_max_f32 v12, 0, v12\n\t" \
"v_mfma_f32_16x16x32_f16 a[56:59], v[112:115], v[192:195], a[56:59]\n\t" \
"v_max_f32 v13, 0, v13\n\t" \
"v_max_f32 v14, 0, v14\n\t" \
"v_max_f32 v15, 0, v15\n\t" \
"buffer_store_dwordx4 v[12:15], %[ooff], s[44:47], s65 offen offset:256 \n\t" \
"v_accvgpr_read_b32 v16, a17\n\t" \
"v_accvgpr_read_b32 v17, a21\n\t" \
"v_mfma_f32_16x16x32_f16 a[56:59], v[80:83], v[188:191], a[56:59]\n\t" \
"v_accvgpr_read_b32 v18, a25\n\t" \
"v_accvgpr_read_b32 v19, a29\n\t" \
"s_mov_b32 s63, 131072\n\t" \
"v_max_f32 v16, 0, v16\n\t" \
"v_max_f32 v17, 0, v17\n\t" \
"v_max_f32 v18, 0, v18\n\t" \
"v_mfma_f32_16x16x32_f16 a[52:55], v[80:83], v[180:183], a[52:55]\n\t" \
"v_max_f32 v19, 0, v19\n\t" \
"buffer_store_dwordx4 v[16:19], %[ooff], s[44:47], s63 offen offset:256 \n\t" \
"v_accvgpr_read_b32 v20, a18\n\t" \
"v_accvgpr_read_b32 v21, a22\n\t" \
"v_accvgpr_read_b32 v22, a26\n\t" \
"v_accvgpr_read_b32 v23, a30\n\t" \
"v_mfma_f32_16x16x32_f16 a[52:55], v[112:115], v[184:187], a[52:55]\n\t" \
"s_mov_b32 s65, 262144\n\t" \
"v_max_f32 v20, 0, v20\n\t" \
"v_max_f32 v21, 0, v21\n\t" \
"v_max_f32 v22, 0, v22\n\t" \
"v_max_f32 v23, 0, v23\n\t" \
"buffer_store_dwordx4 v[20:23], %[ooff], s[44:47], s65 offen offset:256 \n\t" \
"v_mfma_f32_16x16x32_f16 a[48:51], v[112:115], v[176:179], a[48:51]\n\t" \
"v_accvgpr_read_b32 v24, a19\n\t" \
"v_accvgpr_read_b32 v25, a23\n\t" \
"v_accvgpr_read_b32 v26, a27\n\t" \
"v_accvgpr_read_b32 v27, a31\n\t" \
"s_mov_b32 s63, 393216\n\t" \
"v_max_f32 v24, 0, v24\n\t" \
"v_mfma_f32_16x16x32_f16 a[48:51], v[80:83], v[172:175], a[48:51]\n\t" \
"v_max_f32 v25, 0, v25\n\t" \
"v_max_f32 v26, 0, v26\n\t" \
"v_max_f32 v27, 0, v27\n\t" \
"buffer_store_dwordx4 v[24:27], %[ooff], s[44:47], s63 offen offset:256 \n\t" \
"v_mfma_f32_16x16x32_f16 a[80:83], v[84:87], v[172:175], a[80:83]\n\t" \
"v_mfma_f32_16x16x32_f16 a[80:83], v[116:119], v[176:179], a[80:83]\n\t" \
"s_nop 5\n\t" \
"v_accvgpr_read_b32 v28, a48\n\t" \
"v_accvgpr_read_b32 v29, a52\n\t" \
"v_accvgpr_read_b32 v30, a56\n\t" \
"v_accvgpr_read_b32 v31, a60\n\t" \
"s_mov_b32 s65, 2097152\n\t" \
"v_max_f32 v28, 0, v28\n\t" \
"v_mfma_f32_16x16x32_f16 a[84:87], v[116:119], v[184:187], a[84:87]\n\t" \
"v_max_f32 v29, 0, v29\n\t" \
"v_max_f32 v30, 0, v30\n\t" \
"v_max_f32 v31, 0, v31\n\t" \
"buffer_store_dwordx4 v[28:31], %[ooff], s[44:47], s65 offen offset:256 \n\t" \
"v_accvgpr_read_b32 v32, a49\n\t" \
"v_accvgpr_read_b32 v33, a53\n\t" \
"v_mfma_f32_16x16x32_f16 a[84:87], v[84:87], v[180:183], a[84:87]\n\t" \
"v_accvgpr_read_b32 v34, a57\n\t" \
"v_accvgpr_read_b32 v35, a61\n\t" \
"s_mov_b32 s63, 2228224\n\t" \
"v_max_f32 v32, 0, v32\n\t" \
"v_max_f32 v33, 0, v33\n\t" \
"v_max_f32 v34, 0, v34\n\t" \
"v_mfma_f32_16x16x32_f16 a[88:91], v[84:87], v[188:191], a[88:91]\n\t" \
"v_max_f32 v35, 0, v35\n\t" \
"buffer_store_dwordx4 v[32:35], %[ooff], s[44:47], s63 offen offset:256 \n\t" \
"v_accvgpr_read_b32 v36, a50\n\t" \
"v_accvgpr_read_b32 v37, a54\n\t" \
"v_accvgpr_read_b32 v38, a58\n\t" \
"v_accvgpr_read_b32 v39, a62\n\t" \
"v_mfma_f32_16x16x32_f16 a[88:91], v[116:119], v[192:195], a[88:91]\n\t" \
"s_mov_b32 s65, 2359296\n\t" \
"v_max_f32 v36, 0, v36\n\t" \
"v_max_f32 v37, 0, v37\n\t" \
"v_max_f32 v38, 0, v38\n\t" \
"v_max_f32 v39, 0, v39\n\t" \
"buffer_store_dwordx4 v[36:39], %[ooff], s[44:47], s65 offen offset:256 \n\t" \
"v_mfma_f32_16x16x32_f16 a[92:95], v[116:119], v[200:203], a[92:95]\n\t" \
"v_accvgpr_read_b32 v40, a51\n\t" \
"v_accvgpr_read_b32 v41, a55\n\t" \
"v_accvgpr_read_b32 v42, a59\n\t" \
"v_accvgpr_read_b32 v43, a63\n\t" \
"s_mov_b32 s63, 2490368\n\t" \
"v_max_f32 v40, 0, v40\n\t" \
"v_mfma_f32_16x16x32_f16 a[92:95], v[84:87], v[196:199], a[92:95]\n\t" \
"v_max_f32 v41, 0, v41\n\t" \
"v_max_f32 v42, 0, v42\n\t" \
"v_max_f32 v43, 0, v43\n\t" \
"buffer_store_dwordx4 v[40:43], %[ooff], s[44:47], s63 offen offset:256 \n\t" \
"v_mfma_f32_16x16x32_f16 a[124:127], v[88:91], v[196:199], a[124:127]\n\t" \
"v_mfma_f32_16x16x32_f16 a[124:127], v[120:123], v[200:203], a[124:127]\n\t" \
"s_nop 5\n\t" \
"v_accvgpr_read_b32 v44, a80\n\t" \
"v_accvgpr_read_b32 v45, a84\n\t" \
"v_accvgpr_read_b32 v46, a88\n\t" \
"v_accvgpr_read_b32 v47, a92\n\t" \
"s_mov_b32 s65, 4194304\n\t" \
"v_max_f32 v44, 0, v44\n\t" \
"v_mfma_f32_16x16x32_f16 a[120:123], v[120:123], v[192:195], a[120:123]\n\t" \
"v_max_f32 v45, 0, v45\n\t" \
"v_max_f32 v46, 0, v46\n\t" \
"v_max_f32 v47, 0, v47\n\t" \
"buffer_store_dwordx4 v[44:47], %[ooff], s[44:47], s65 offen offset:256 \n\t" \
"v_accvgpr_read_b32 v48, a81\n\t" \
"v_accvgpr_read_b32 v49, a85\n\t" \
"v_mfma_f32_16x16x32_f16 a[120:123], v[88:91], v[188:191], a[120:123]\n\t" \
"v_accvgpr_read_b32 v50, a89\n\t" \
"v_accvgpr_read_b32 v51, a93\n\t" \
"s_mov_b32 s63, 4325376\n\t" \
"v_max_f32 v48, 0, v48\n\t" \
"v_max_f32 v49, 0, v49\n\t" \
"v_max_f32 v50, 0, v50\n\t" \
"v_mfma_f32_16x16x32_f16 a[116:119], v[88:91], v[180:183], a[116:119]\n\t" \
"v_max_f32 v51, 0, v51\n\t" \
"buffer_store_dwordx4 v[48:51], %[ooff], s[44:47], s63 offen offset:256 \n\t" \
"v_accvgpr_read_b32 v52, a82\n\t" \
"v_accvgpr_read_b32 v53, a86\n\t" \
"v_accvgpr_read_b32 v54, a90\n\t" \
"v_accvgpr_read_b32 v55, a94\n\t" \
"v_mfma_f32_16x16x32_f16 a[116:119], v[120:123], v[184:187], a[116:119]\n\t" \
"s_mov_b32 s65, 4456448\n\t" \
"v_max_f32 v52, 0, v52\n\t" \
"v_max_f32 v53, 0, v53\n\t" \
"v_max_f32 v54, 0, v54\n\t" \
"v_max_f32 v55, 0, v55\n\t" \
"buffer_store_dwordx4 v[52:55], %[ooff], s[44:47], s65 offen offset:256 \n\t" \
"v_mfma_f32_16x16x32_f16 a[112:115], v[120:123], v[176:179], a[112:115]\n\t" \
"v_accvgpr_read_b32 v56, a83\n\t" \
"v_accvgpr_read_b32 v57, a87\n\t" \
"v_accvgpr_read_b32 v58, a91\n\t" \
"v_accvgpr_read_b32 v59, a95\n\t" \
"s_mov_b32 s63, 4587520\n\t" \
"v_max_f32 v56, 0, v56\n\t" \
"v_mfma_f32_16x16x32_f16 a[112:115], v[88:91], v[172:175], a[112:115]\n\t" \
"v_max_f32 v57, 0, v57\n\t" \
"v_max_f32 v58, 0, v58\n\t" \
"v_max_f32 v59, 0, v59\n\t" \
"buffer_store_dwordx4 v[56:59], %[ooff], s[44:47], s63 offen offset:256 \n\t" \
"v_mfma_f32_16x16x32_f16 a[144:147], v[92:95], v[172:175], a[144:147]\n\t" \
"v_mfma_f32_16x16x32_f16 a[144:147], v[124:127], v[176:179], a[144:147]\n\t" \
"s_nop 5\n\t" \
"v_accvgpr_read_b32 v60, a112\n\t" \
"v_accvgpr_read_b32 v61, a116\n\t" \
"v_accvgpr_read_b32 v62, a120\n\t" \
"v_accvgpr_read_b32 v63, a124\n\t" \
"s_mov_b32 s65, 6291456\n\t" \
"v_max_f32 v60, 0, v60\n\t" \
"v_mfma_f32_16x16x32_f16 a[148:151], v[124:127], v[184:187], a[148:151]\n\t" \
"v_max_f32 v61, 0, v61\n\t" \
"v_max_f32 v62, 0, v62\n\t" \
"v_max_f32 v63, 0, v63\n\t" \
"buffer_store_dwordx4 v[60:63], %[ooff], s[44:47], s65 offen offset:256 \n\t" \
"v_accvgpr_read_b32 v64, a113\n\t" \
"v_accvgpr_read_b32 v65, a117\n\t" \
"v_mfma_f32_16x16x32_f16 a[148:151], v[92:95], v[180:183], a[148:151]\n\t" \
"v_accvgpr_read_b32 v66, a121\n\t" \
"v_accvgpr_read_b32 v67, a125\n\t" \
"s_mov_b32 s63, 6422528\n\t" \
"v_max_f32 v64, 0, v64\n\t" \
"v_max_f32 v65, 0, v65\n\t" \
"v_max_f32 v66, 0, v66\n\t" \
"v_mfma_f32_16x16x32_f16 a[152:155], v[92:95], v[188:191], a[152:155]\n\t" \
"v_max_f32 v67, 0, v67\n\t" \
"buffer_store_dwordx4 v[64:67], %[ooff], s[44:47], s63 offen offset:256 \n\t" \
"v_accvgpr_read_b32 v68, a114\n\t" \
"v_accvgpr_read_b32 v69, a118\n\t" \
"v_accvgpr_read_b32 v70, a122\n\t" \
"v_accvgpr_read_b32 v71, a126\n\t" \
"v_mfma_f32_16x16x32_f16 a[152:155], v[124:127], v[192:195], a[152:155]\n\t" \
"s_mov_b32 s65, 6553600\n\t" \
"v_max_f32 v68, 0, v68\n\t" \
"v_max_f32 v69, 0, v69\n\t" \
"v_max_f32 v70, 0, v70\n\t" \
"v_max_f32 v71, 0, v71\n\t" \
"buffer_store_dwordx4 v[68:71], %[ooff], s[44:47], s65 offen offset:256 \n\t" \
"v_mfma_f32_16x16x32_f16 a[156:159], v[124:127], v[200:203], a[156:159]\n\t" \
"v_accvgpr_read_b32 v72, a115\n\t" \
"v_accvgpr_read_b32 v73, a119\n\t" \
"v_accvgpr_read_b32 v74, a123\n\t" \
"v_accvgpr_read_b32 v75, a127\n\t" \
"s_mov_b32 s63, 6684672\n\t" \
"v_max_f32 v72, 0, v72\n\t" \
"v_mfma_f32_16x16x32_f16 a[156:159], v[92:95], v[196:199], a[156:159]\n\t" \
"v_max_f32 v73, 0, v73\n\t" \
"v_max_f32 v74, 0, v74\n\t" \
"v_max_f32 v75, 0, v75\n\t" \
"buffer_store_dwordx4 v[72:75], %[ooff], s[44:47], s63 offen offset:256 \n\t" \
"v_mfma_f32_16x16x32_f16 a[188:191], v[96:99], v[196:199], a[188:191]\n\t" \
"v_mfma_f32_16x16x32_f16 a[188:191], v[128:131], v[200:203], a[188:191]\n\t" \
"s_nop 5\n\t" \
"v_accvgpr_read_b32 v12, a144\n\t" \
"v_accvgpr_read_b32 v13, a148\n\t" \
"v_accvgpr_read_b32 v14, a152\n\t" \
"v_accvgpr_read_b32 v15, a156\n\t" \
"s_mov_b32 s65, 8388608\n\t" \
"v_max_f32 v12, 0, v12\n\t" \
"v_mfma_f32_16x16x32_f16 a[184:187], v[128:131], v[192:195], a[184:187]\n\t" \
"v_max_f32 v13, 0, v13\n\t" \
"v_max_f32 v14, 0, v14\n\t" \
"v_max_f32 v15, 0, v15\n\t" \
"buffer_store_dwordx4 v[12:15], %[ooff], s[44:47], s65 offen offset:256 \n\t" \
"v_accvgpr_read_b32 v16, a145\n\t" \
"v_accvgpr_read_b32 v17, a149\n\t" \
"v_mfma_f32_16x16x32_f16 a[184:187], v[96:99], v[188:191], a[184:187]\n\t" \
"v_accvgpr_read_b32 v18, a153\n\t" \
"v_accvgpr_read_b32 v19, a157\n\t" \
"s_mov_b32 s63, 8519680\n\t" \
"v_max_f32 v16, 0, v16\n\t" \
"v_max_f32 v17, 0, v17\n\t" \
"v_max_f32 v18, 0, v18\n\t" \
"v_mfma_f32_16x16x32_f16 a[180:183], v[96:99], v[180:183], a[180:183]\n\t" \
"v_max_f32 v19, 0, v19\n\t" \
"buffer_store_dwordx4 v[16:19], %[ooff], s[44:47], s63 offen offset:256 \n\t" \
"v_accvgpr_read_b32 v20, a146\n\t" \
"v_accvgpr_read_b32 v21, a150\n\t" \
"v_accvgpr_read_b32 v22, a154\n\t" \
"v_accvgpr_read_b32 v23, a158\n\t" \
"v_mfma_f32_16x16x32_f16 a[180:183], v[128:131], v[184:187], a[180:183]\n\t" \
"s_mov_b32 s65, 8650752\n\t" \
"v_max_f32 v20, 0, v20\n\t" \
"v_max_f32 v21, 0, v21\n\t" \
"v_max_f32 v22, 0, v22\n\t" \
"v_max_f32 v23, 0, v23\n\t" \
"buffer_store_dwordx4 v[20:23], %[ooff], s[44:47], s65 offen offset:256 \n\t" \
"v_mfma_f32_16x16x32_f16 a[176:179], v[128:131], v[176:179], a[176:179]\n\t" \
"v_accvgpr_read_b32 v24, a147\n\t" \
"v_accvgpr_read_b32 v25, a151\n\t" \
"v_accvgpr_read_b32 v26, a155\n\t" \
"v_accvgpr_read_b32 v27, a159\n\t" \
"s_mov_b32 s63, 8781824\n\t" \
"v_max_f32 v24, 0, v24\n\t" \
"v_mfma_f32_16x16x32_f16 a[176:179], v[96:99], v[172:175], a[176:179]\n\t" \
"v_max_f32 v25, 0, v25\n\t" \
"v_max_f32 v26, 0, v26\n\t" \
"v_max_f32 v27, 0, v27\n\t" \
"buffer_store_dwordx4 v[24:27], %[ooff], s[44:47], s63 offen offset:256 \n\t" \
"v_mfma_f32_16x16x32_f16 a[208:211], v[100:103], v[172:175], a[208:211]\n\t" \
"v_mfma_f32_16x16x32_f16 a[208:211], v[132:135], v[176:179], a[208:211]\n\t" \
"s_nop 5\n\t" \
"v_accvgpr_read_b32 v28, a176\n\t" \
"v_accvgpr_read_b32 v29, a180\n\t" \
"v_accvgpr_read_b32 v30, a184\n\t" \
"v_accvgpr_read_b32 v31, a188\n\t" \
"s_mov_b32 s65, 10485760\n\t" \
"v_max_f32 v28, 0, v28\n\t" \
"v_mfma_f32_16x16x32_f16 a[212:215], v[132:135], v[184:187], a[212:215]\n\t" \
"v_max_f32 v29, 0, v29\n\t" \
"v_max_f32 v30, 0, v30\n\t" \
"v_max_f32 v31, 0, v31\n\t" \
"buffer_store_dwordx4 v[28:31], %[ooff], s[44:47], s65 offen offset:256 \n\t" \
"v_accvgpr_read_b32 v32, a177\n\t" \
"v_accvgpr_read_b32 v33, a181\n\t" \
"v_mfma_f32_16x16x32_f16 a[212:215], v[100:103], v[180:183], a[212:215]\n\t" \
"v_accvgpr_read_b32 v34, a185\n\t" \
"v_accvgpr_read_b32 v35, a189\n\t" \
"s_mov_b32 s63, 10616832\n\t" \
"v_max_f32 v32, 0, v32\n\t" \
"v_max_f32 v33, 0, v33\n\t" \
"v_max_f32 v34, 0, v34\n\t" \
"v_mfma_f32_16x16x32_f16 a[216:219], v[100:103], v[188:191], a[216:219]\n\t" \
"v_max_f32 v35, 0, v35\n\t" \
"buffer_store_dwordx4 v[32:35], %[ooff], s[44:47], s63 offen offset:256 \n\t" \
"v_accvgpr_read_b32 v36, a178\n\t" \
"v_accvgpr_read_b32 v37, a182\n\t" \
"v_accvgpr_read_b32 v38, a186\n\t" \
"v_accvgpr_read_b32 v39, a190\n\t" \
"v_mfma_f32_16x16x32_f16 a[216:219], v[132:135], v[192:195], a[216:219]\n\t" \
"s_mov_b32 s65, 10747904\n\t" \
"v_max_f32 v36, 0, v36\n\t" \
"v_max_f32 v37, 0, v37\n\t" \
"v_max_f32 v38, 0, v38\n\t" \
"v_max_f32 v39, 0, v39\n\t" \
"buffer_store_dwordx4 v[36:39], %[ooff], s[44:47], s65 offen offset:256 \n\t" \
"v_mfma_f32_16x16x32_f16 a[220:223], v[132:135], v[200:203], a[220:223]\n\t" \
"v_accvgpr_read_b32 v40, a179\n\t" \
"v_accvgpr_read_b32 v41, a183\n\t" \
"v_accvgpr_read_b32 v42, a187\n\t" \
"v_accvgpr_read_b32 v43, a191\n\t" \
"s_mov_b32 s63, 10878976\n\t" \
"v_max_f32 v40, 0, v40\n\t" \
"v_mfma_f32_16x16x32_f16 a[220:223], v[100:103], v[196:199], a[220:223]\n\t" \
"v_max_f32 v41, 0, v41\n\t" \
"v_max_f32 v42, 0, v42\n\t" \
"v_max_f32 v43, 0, v43\n\t" \
"buffer_store_dwordx4 v[40:43], %[ooff], s[44:47], s63 offen offset:256 \n\t" \
"v_mfma_f32_16x16x32_f16 a[252:255], v[104:107], v[196:199], a[252:255]\n\t" \
"v_mfma_f32_16x16x32_f16 a[252:255], v[136:139], v[200:203], a[252:255]\n\t" \
"s_nop 5\n\t" \
"v_accvgpr_read_b32 v44, a208\n\t" \
"v_accvgpr_read_b32 v45, a212\n\t" \
"v_accvgpr_read_b32 v46, a216\n\t" \
"v_accvgpr_read_b32 v47, a220\n\t" \
"s_mov_b32 s65, 12582912\n\t" \
"v_max_f32 v44, 0, v44\n\t" \
"v_mfma_f32_16x16x32_f16 a[248:251], v[136:139], v[192:195], a[248:251]\n\t" \
"v_max_f32 v45, 0, v45\n\t" \
"v_max_f32 v46, 0, v46\n\t" \
"v_max_f32 v47, 0, v47\n\t" \
"buffer_store_dwordx4 v[44:47], %[ooff], s[44:47], s65 offen offset:256 \n\t" \
"v_accvgpr_read_b32 v48, a209\n\t" \
"v_accvgpr_read_b32 v49, a213\n\t" \
"v_mfma_f32_16x16x32_f16 a[248:251], v[104:107], v[188:191], a[248:251]\n\t" \
"v_accvgpr_read_b32 v50, a217\n\t" \
"v_accvgpr_read_b32 v51, a221\n\t" \
"s_mov_b32 s63, 12713984\n\t" \
"v_max_f32 v48, 0, v48\n\t" \
"v_max_f32 v49, 0, v49\n\t" \
"v_max_f32 v50, 0, v50\n\t" \
"v_mfma_f32_16x16x32_f16 a[244:247], v[104:107], v[180:183], a[244:247]\n\t" \
"v_max_f32 v51, 0, v51\n\t" \
"buffer_store_dwordx4 v[48:51], %[ooff], s[44:47], s63 offen offset:256 \n\t" \
"v_accvgpr_read_b32 v52, a210\n\t" \
"v_accvgpr_read_b32 v53, a214\n\t" \
"v_accvgpr_read_b32 v54, a218\n\t" \
"v_accvgpr_read_b32 v55, a222\n\t" \
"v_mfma_f32_16x16x32_f16 a[244:247], v[136:139], v[184:187], a[244:247]\n\t" \
"s_mov_b32 s65, 12845056\n\t" \
"v_max_f32 v52, 0, v52\n\t" \
"v_max_f32 v53, 0, v53\n\t" \
"v_max_f32 v54, 0, v54\n\t" \
"v_max_f32 v55, 0, v55\n\t" \
"buffer_store_dwordx4 v[52:55], %[ooff], s[44:47], s65 offen offset:256 \n\t" \
"v_mfma_f32_16x16x32_f16 a[240:243], v[136:139], v[176:179], a[240:243]\n\t" \
"v_accvgpr_read_b32 v56, a211\n\t" \
"v_accvgpr_read_b32 v57, a215\n\t" \
"v_accvgpr_read_b32 v58, a219\n\t" \
"v_accvgpr_read_b32 v59, a223\n\t" \
"s_mov_b32 s63, 12976128\n\t" \
"v_max_f32 v56, 0, v56\n\t" \
"v_mfma_f32_16x16x32_f16 a[240:243], v[104:107], v[172:175], a[240:243]\n\t" \
"v_max_f32 v57, 0, v57\n\t" \
"v_max_f32 v58, 0, v58\n\t" \
"v_max_f32 v59, 0, v59\n\t" \
"buffer_store_dwordx4 v[56:59], %[ooff], s[44:47], s63 offen offset:256 \n\t" \
"s_nop 9\n\t" \
"v_accvgpr_read_b32 v60, a240\n\t" \
"v_accvgpr_read_b32 v61, a244\n\t" \
"v_accvgpr_read_b32 v62, a248\n\t" \
"v_accvgpr_read_b32 v63, a252\n\t" \
"s_mov_b32 s65, 14680064\n\t" \
"v_max_f32 v60, 0, v60\n\t" \
"v_max_f32 v61, 0, v61\n\t" \
"v_max_f32 v62, 0, v62\n\t" \
"v_max_f32 v63, 0, v63\n\t" \
"buffer_store_dwordx4 v[60:63], %[ooff], s[44:47], s65 offen offset:256 \n\t" \
"v_accvgpr_read_b32 v64, a241\n\t" \
"v_accvgpr_read_b32 v65, a245\n\t" \
"v_accvgpr_read_b32 v66, a249\n\t" \
"v_accvgpr_read_b32 v67, a253\n\t" \
"s_mov_b32 s63, 14811136\n\t" \
"v_max_f32 v64, 0, v64\n\t" \
"v_max_f32 v65, 0, v65\n\t" \
"v_max_f32 v66, 0, v66\n\t" \
"v_max_f32 v67, 0, v67\n\t" \
"buffer_store_dwordx4 v[64:67], %[ooff], s[44:47], s63 offen offset:256 \n\t" \
"v_accvgpr_read_b32 v68, a242\n\t" \
"v_accvgpr_read_b32 v69, a246\n\t" \
"v_accvgpr_read_b32 v70, a250\n\t" \
"v_accvgpr_read_b32 v71, a254\n\t" \
"s_mov_b32 s65, 14942208\n\t" \
"v_max_f32 v68, 0, v68\n\t" \
"v_max_f32 v69, 0, v69\n\t" \
"v_max_f32 v70, 0, v70\n\t" \
"v_max_f32 v71, 0, v71\n\t" \
"buffer_store_dwordx4 v[68:71], %[ooff], s[44:47], s65 offen offset:256 \n\t" \
"v_accvgpr_read_b32 v72, a243\n\t" \
"v_accvgpr_read_b32 v73, a247\n\t" \
"v_accvgpr_read_b32 v74, a251\n\t" \
"v_accvgpr_read_b32 v75, a255\n\t" \
"s_mov_b32 s63, 15073280\n\t" \
"v_max_f32 v72, 0, v72\n\t" \
"v_max_f32 v73, 0, v73\n\t" \
"v_max_f32 v74, 0, v74\n\t" \
"v_max_f32 v75, 0, v75\n\t" \
"buffer_store_dwordx4 v[72:75], %[ooff], s[44:47], s63 offen offset:256 \n\t" \
"s_branch KEND_%=\n\t" \
"NANOUT_%=:\n\t" \
"s_waitcnt vmcnt(0) lgkmcnt(0)\n\t" \
"v_mov_b32 v12, 0x7fc00000\n\t" \
"v_mov_b32 v13, 0x7fc00000\n\t" \
"v_mov_b32 v14, 0x7fc00000\n\t" \
"v_mov_b32 v15, 0x7fc00000\n\t" \
"s_mov_b32 s63, 0\n\t" \
"NANL_%=:\n\t" \
"s_lshr_b32 s65, s63, 2\n\t" \
"s_lshl_b32 s65, s65, 4\n\t" \
"s_and_b32 s66, s63, 3\n\t" \
"s_add_u32 s65, s65, s66\n\t" \
"s_lshl_b32 s65, s65, 17\n\t" \
"buffer_store_dwordx4 v[12:15], %[ooff], s[44:47], s65 offen\n\t" \
"buffer_store_dwordx4 v[12:15], %[ooff], s[44:47], s65 offen offset:256\n\t" \
"s_add_u32 s63, s63, 1\n\t" \
"s_cmp_lt_u32 s63, 32\n\t" \
"s_cbranch_scc1 NANL_%=\n\t" \
"s_branch KEND_%=\n\t" \
"SPIN0_%=:\n\t" \
"buffer_load_dword v250, %[uid4], s[48:51], 0 offen offset:1024 sc1\n\t" \
"s_memrealtime s[72:73]\n\t" \
"s_waitcnt vmcnt(0) lgkmcnt(0)\n\t" \
"v_cmp_le_u32 vcc, s67, v250\n\t" \
"v_readlane_b32 s76, v250, 32\n\t" \
"s_nop 3\n\t" \
"s_and_b32 s65, vcc_lo, 15\n\t" \
"s_cmp_eq_u32 s65, 15\n\t" \
"s_cbranch_scc1 SPIN0RET_%=\n\t" \
"s_cmp_eq_u32 s76, s68\n\t" \
"s_cbranch_scc1 SPIN0F_%=\n\t" \
"s_sub_u32 s72, s72, s70\n\t" \
"s_subb_u32 s73, s73, s71\n\t" \
"s_cmp_lg_u32 s73, 0\n\t" \
"s_cbranch_scc1 SPIN0F_%=\n\t" \
"s_cmp_gt_u32 s72, 100000000\n\t" \
"s_cbranch_scc1 SPIN0F_%=\n\t" \
"s_sleep 4\n\t" \
"s_branch SPIN0_%=\n\t" \
"SPIN0F_%=:\n\t" \
"s_mov_b32 s69, 256\n\t" \
"v_mov_b32 v251, s68\n\t" \
"s_mov_b64 exec, 1\n\t" \
"buffer_store_dword v251, off, s[48:51], 0 offset:1152 sc1\n\t" \
"s_mov_b64 exec, -1\n\t" \
"s_branch SPIN0RET_%=\n\t" \
"SPINL_%=:\n\t" \
"buffer_load_dword v250, %[poff], s[48:51], 0 offen sc1\n\t" \
"buffer_load_dword v254, off, s[48:51], 0 offset:1152 sc1\n\t" \
"s_memrealtime s[72:73]\n\t" \
"s_waitcnt vmcnt(0) lgkmcnt(0)\n\t" \
"v_max_u32 v253, v253, v250\n\t" \
"v_cmp_le_u32 vcc, s67, v250\n\t" \
"v_readfirstlane_b32 s76, v254\n\t" \
"v_readlane_b32 s63, v253, s64\n\t" \
"s_nop 3\n\t" \
"s_andn2_b32 s65, vcc_hi, s79\n\t" \
"s_cbranch_scc0 SPINL2_%=\n\t" \
"s_mov_b32 exec_lo, s65\n\t" \
"s_mov_b32 exec_hi, 0\n\t" \
"v_mbcnt_lo_u32_b32 v251, s65, 0\n\t" \
"v_mbcnt_lo_u32_b32 v254, -1, 0\n\t" \
"v_add_u32 v251, s88, v251\n\t" \
"v_add_u32 v254, s87, v254\n\t" \
"v_lshlrev_b32 v251, 2, v251\n\t" \
"buffer_store_dword v254, v251, s[48:51], 0 offen offset:1280 sc0 sc1\n\t" \
"s_mov_b64 exec, -1\n\t" \
"s_or_b32 s79, s79, s65\n\t" \
"s_bcnt1_i32_b32 s65, s65\n\t" \
"s_add_u32 s88, s88, s65\n\t" \
"SPINL2_%=:\n\t" \
"s_cmp_ge_u32 s63, s87\n\t" \
"s_cbranch_scc1 DECRET_%=\n\t" \
"s_cmp_eq_u32 s76, s68\n\t" \
"s_cbranch_scc1 SPINLF_%=\n\t" \
"s_sub_u32 s72, s72, s70\n\t" \
"s_subb_u32 s73, s73, s71\n\t" \
"s_cmp_lg_u32 s73, 0\n\t" \
"s_cbranch_scc1 SPINLF_%=\n\t" \
"s_cmp_gt_u32 s72, 100000000\n\t" \
"s_cbranch_scc1 SPINLF_%=\n\t" \
"s_sleep 4\n\t" \
"s_branch SPINL_%=\n\t" \
"SPINLF_%=:\n\t" \
"s_mov_b32 s69, 256\n\t" \
"v_mov_b32 v251, s68\n\t" \
"s_mov_b64 exec, 1\n\t" \
"buffer_store_dword v251, off, s[48:51], 0 offset:1152 sc1\n\t" \
"s_mov_b64 exec, -1\n\t" \
"s_mov_b32 s63, s64\n\t" \
"s_branch DECRET_%=\n\t" \
"APPEND_%=:\n\t" \
"s_mov_b32 exec_lo, s65\n\t" \
"s_mov_b32 exec_hi, 0\n\t" \
"v_mbcnt_lo_u32_b32 v251, s65, 0\n\t" \
"v_mbcnt_lo_u32_b32 v254, -1, 0\n\t" \
"v_add_u32 v251, s88, v251\n\t" \
"v_add_u32 v254, s87, v254\n\t" \
"v_lshlrev_b32 v251, 2, v251\n\t" \
"buffer_store_dword v254, v251, s[48:51], 0 offen offset:1280 sc0 sc1\n\t" \
"s_mov_b64 exec, -1\n\t" \
"s_or_b32 s79, s79, s65\n\t" \
"s_bcnt1_i32_b32 s65, s65\n\t" \
"s_add_u32 s88, s88, s65\n\t" \
"s_nop 1\n\t" \
"s_branch APPRET_%=\n\t" \
"PUSHB_%=:\n\t" \
"ds_read_b128 v[236:239], %[ldsstg] offset:0\n\t" \
"ds_read_b128 v[240:243], %[ldsstg] offset:4096\n\t" \
"s_waitcnt lgkmcnt(0)\n\t" \
"s_add_u32 s63, s78, 0\n\t" \
"buffer_store_dwordx4 v[236:239], %[pshoff], s[52:55], s63 offen sc0 sc1\n\t" \
"s_add_u32 s63, s78, 1024\n\t" \
"buffer_store_dwordx4 v[240:243], %[pshoff], s[52:55], s63 offen sc0 sc1\n\t" \
"s_nop 2\n\t" \
"ds_read_b128 v[236:239], %[ldsstg] offset:8192\n\t" \
"ds_read_b128 v[240:243], %[ldsstg] offset:12288\n\t" \
"s_waitcnt lgkmcnt(0)\n\t" \
"s_add_u32 s63, s78, 16384\n\t" \
"buffer_store_dwordx4 v[236:239], %[pshoff], s[52:55], s63 offen sc0 sc1\n\t" \
"s_add_u32 s63, s78, 17408\n\t" \
"buffer_store_dwordx4 v[240:243], %[pshoff], s[52:55], s63 offen sc0 sc1\n\t" \
"s_nop 2\n\t" \
"s_branch PUSHRET_%=\n\t" \
"FLAGB_%=:\n\t" \
"v_mov_b32 v250, 1\n\t" \
"s_mov_b64 exec, 1\n\t" \
"buffer_atomic_add v250, off, s[52:55], %[flgoff] sc1\n\t" \
"s_mov_b64 exec, -1\n\t" \
"s_nop 1\n\t" \
"s_branch FLAGRET_%=\n\t" \
"KEND_%=:\n\t" \
"s_cmp_lg_u32 %[wave], 0\n\t" \
"s_cbranch_scc1 KEND2_%=\n\t" \
"s_memrealtime s[72:73]\n\t" \
"v_mov_b32 v236, s68\n\t" \
"s_waitcnt lgkmcnt(0)\n\t" \
"v_mov_b32 v237, s72\n\t" \
"s_mov_b64 exec, 1\n\t" \
"buffer_store_dwordx2 v[236:237], off, s[48:51], %[ctroff] sc1\n\t" \
"s_mov_b64 exec, -1\n\t" \
"KEND2_%=:\n\t" \
"s_waitcnt vmcnt(0)\n\t" \
"s_waitcnt vmcnt(0)\n\t" \
"KLAST_%=:\n\t" \
""
#define GEMM_CLOBBERS "v12", "v13", "v14", "v15", "v16", "v17", "v18", "v19", "v20", "v21", "v22", "v23", "v24", "v25", "v26", "v27", "v28", "v29", "v30", "v31", "v32", "v33", "v34", "v35", "v36", "v37", "v38", "v39", "v40", "v41", "v42", "v43", "v44", "v45", "v46", "v47", "v48", "v49", "v50", "v51", "v52", "v53", "v54", "v55", "v56", "v57", "v58", "v59", "v60", "v61", "v62", "v63", "v64", "v65", "v66", "v67", "v68", "v69", "v70", "v71", "v72", "v73", "v74", "v75", "v76", "v77", "v78", "v79", "v80", "v81", "v82", "v83", "v84", "v85", "v86", "v87", "v88", "v89", "v90", "v91", "v92", "v93", "v94", "v95", "v96", "v97", "v98", "v99", "v100", "v101", "v102", "v103", "v104", "v105", "v106", "v107", "v108", "v109", "v110", "v111", "v112", "v113", "v114", "v115", "v116", "v117", "v118", "v119", "v120", "v121", "v122", "v123", "v124", "v125", "v126", "v127", "v128", "v129", "v130", "v131", "v132", "v133", "v134", "v135", "v136", "v137", "v138", "v139", "v140", "v141", "v142", "v143", "v144", "v145", "v146", "v147", "v148", "v149", "v150", "v151", "v152", "v153", "v154", "v155", "v156", "v157", "v158", "v159", "v160", "v161", "v162", "v163", "v164", "v165", "v166", "v167", "v168", "v169", "v170", "v171", "v172", "v173", "v174", "v175", "v176", "v177", "v178", "v179", "v180", "v181", "v182", "v183", "v184", "v185", "v186", "v187", "v188", "v189", "v190", "v191", "v192", "v193", "v194", "v195", "v196", "v197", "v198", "v199", "v200", "v201", "v202", "v203", "v204", "v205", "v206", "v207", "v208", "v209", "v210", "v211", "v212", "v213", "v214", "v215", "v216", "v217", "v218", "v219", "v220", "v221", "v222", "v223", "v224", "v225", "v226", "v227", "v228", "v229", "v230", "v231", "v232", "v233", "v234", "v235", "v236", "v237", "v238", "v239", "v240", "v241", "v242", "v243", "v244", "v245", "v246", "v247", "v248", "v249", "v250", "v251", "v252", "v253", "v254", "a0", "a1", "a2", "a3", "a4", "a5", "a6", "a7", "a8", "a9", "a10", "a11", "a12", "a13", "a14", "a15", "a16", "a17", "a18", "a19", "a20", "a21", "a22", "a23", "a24", "a25", "a26", "a27", "a28", "a29", "a30", "a31", "a32", "a33", "a34", "a35", "a36", "a37", "a38", "a39", "a40", "a41", "a42", "a43", "a44", "a45", "a46", "a47", "a48", "a49", "a50", "a51", "a52", "a53", "a54", "a55", "a56", "a57", "a58", "a59", "a60", "a61", "a62", "a63", "a64", "a65", "a66", "a67", "a68", "a69", "a70", "a71", "a72", "a73", "a74", "a75", "a76", "a77", "a78", "a79", "a80", "a81", "a82", "a83", "a84", "a85", "a86", "a87", "a88", "a89", "a90", "a91", "a92", "a93", "a94", "a95", "a96", "a97", "a98", "a99", "a100", "a101", "a102", "a103", "a104", "a105", "a106", "a107", "a108", "a109", "a110", "a111", "a112", "a113", "a114", "a115", "a116", "a117", "a118", "a119", "a120", "a121", "a122", "a123", "a124", "a125", "a126", "a127", "a128", "a129", "a130", "a131", "a132", "a133", "a134", "a135", "a136", "a137", "a138", "a139", "a140", "a141", "a142", "a143", "a144", "a145", "a146", "a147", "a148", "a149", "a150", "a151", "a152", "a153", "a154", "a155", "a156", "a157", "a158", "a159", "a160", "a161", "a162", "a163", "a164", "a165", "a166", "a167", "a168", "a169", "a170", "a171", "a172", "a173", "a174", "a175", "a176", "a177", "a178", "a179", "a180", "a181", "a182", "a183", "a184", "a185", "a186", "a187", "a188", "a189", "a190", "a191", "a192", "a193", "a194", "a195", "a196", "a197", "a198", "a199", "a200", "a201", "a202", "a203", "a204", "a205", "a206", "a207", "a208", "a209", "a210", "a211", "a212", "a213", "a214", "a215", "a216", "a217", "a218", "a219", "a220", "a221", "a222", "a223", "a224", "a225", "a226", "a227", "a228", "a229", "a230", "a231", "a232", "a233", "a234", "a235", "a236", "a237", "a238", "a239", "a240", "a241", "a242", "a243", "a244", "a245", "a246", "a247", "a248", "a249", "a250", "a251", "a252", "a253", "a254", "a255", "s36", "s37", "s38", "s39", "s40", "s41", "s42", "s43", "s44", "s45", "s46", "s47", "s48", "s49", "s50", "s51", "s52", "s53", "s54", "s55", "s56", "s57", "s58", "s59", "s60", "s61", "s62", "s63", "s64", "s65", "s66", "s67", "s68", "s69", "s70", "s71", "s72", "s73", "s74", "s75", "s76", "s77", "s78", "s79", "s80", "s81", "s82", "s83", "s84", "s85", "s86", "s87", "s88", "s89", "s90", "s91", "s92", "s93", "s94", "m0", "scc", "vcc", "memory"

namespace {
constexpr unsigned XS_BYTES = 4u << 20;
constexpr unsigned CTRL_BYTES = 64u << 10;
constexpr unsigned OFF_FLAG = 1024;
constexpr unsigned LDS_STAGE = 131072;
constexpr unsigned LDS_MISC = 131072 + 16384;
constexpr unsigned LDS_BYTES = LDS_MISC + 256;
}

__global__ __launch_bounds__(256, 1) void a2a_gemm_kernel(int rank_i, const float* __restrict__ x,
        const float* __restrict__ W, float* __restrict__ out, void* const* __restrict__ ws_all) {
  extern __shared__ __attribute__((aligned(1024))) char smem[];
  const unsigned tid = threadIdx.x;
  const unsigned lane = tid & 63u;
  const unsigned wave = __builtin_amdgcn_readfirstlane(tid >> 6);
  const unsigned wg = blockIdx.x;
  const unsigned rank = (unsigned)rank_i & 7u;
  const unsigned xcd = wg & 7u, idx = wg >> 3;
  const unsigned n0 = (xcd * 32u + idx) * 128u;
  const unsigned d = (xcd + rank) & 7u;
  const unsigned Pl = idx >> 3, mq = idx & 7u;
  const unsigned slot = (d == rank || Pl == 0u) ? 0u : 1u + (Pl - 1u) * 2u + (mq >> 2);
  const unsigned pushc = slot ? 33u - slot : 0u;
  const unsigned flagc = (slot && wave == 0u) ? 32u - slot : 0u;
  const unsigned uid = rank * 4u + Pl;
  const unsigned dstoff = uid * 131072u + (mq >> 1) * 32768u + (mq & 1u) * 8192u;
  const unsigned flgoff = OFF_FLAG + uid * 4u;
  const unsigned dbgoff = wave == 0u ? 16384u + wg * 160u : 0x20000u;
  const unsigned g = lane >> 4;
  const unsigned xioff = ((d * 512u + (mq * 4u + wave) * 16u + (lane & 15u)) * 512u + Pl * 128u + 64u * (g & 1u) + 32u * (g >> 1)) * 4u;
  const unsigned pshoff = wave * 2048u + lane * 16u;
  unsigned uid4;
  if (lane < 4u) uid4 = (rank * 4u + lane) * 4u;
  else if (lane < 32u) { unsigned j = lane - 4u; uid4 = ((((rank + 1u + j % 7u) & 7u) * 4u) + j / 7u) * 4u; }
  else if (lane == 32u) uid4 = 128u;
  else uid4 = 0x10000u;
  const bool sched = (wg == 8u) && (wave == 0u);
  const unsigned avinit = sched ? 15u : 0xffffffffu;
  const unsigned ctroff = 2048u + wg * 8u;
  unsigned poff;
  {
    const unsigned l2 = lane & 31u;
    unsigned u2;
    if (l2 < 4u) u2 = (rank * 4u + l2) * 4u;
    else { unsigned j = l2 - 4u; u2 = ((((rank + 1u + j % 7u) & 7u) * 4u) + j / 7u) * 4u; }
    poff = lane < 32u ? 1280u + lane * 4u : (sched ? OFF_FLAG + u2 : 0x20000u);
  }
  const unsigned ldsbase = (unsigned)(size_t)smem;
  const unsigned h = lane >> 5, c = lane & 31u;
  const unsigned woff = (h * 32u + wave * 4u) * 131072u + n0 * 4u + c * 16u;
  const unsigned ldsrd = ldsbase + lane * 16u;
  const unsigned ldswr = ldsbase + wave * 8192u + (c >> 4) * 4096u + (2u * h) * 256u + (c & 15u) * 16u;
  const unsigned xoff = wave * 32768u + lane * 16u;
  const unsigned pfoff = (idx * 4u + wave) * 1024u + lane * 16u;
  const unsigned ooff = (wave * 128u + (lane >> 4) * 4u) * 131072u + (n0 + (lane & 15u) * 4u) * 4u;
  const unsigned ldsstg = ldsbase + LDS_STAGE + wave * 1024u + lane * 16u;
  const unsigned ldsdec = ldsbase + LDS_MISC + wave * 4u;
  const unsigned ldsd0 = ldsbase + LDS_MISC;
  const unsigned long long wp = (unsigned long long)W, op = (unsigned long long)out, xi = (unsigned long long)x;
  const unsigned long long mp = (unsigned long long)ws_all[rank], dp = (unsigned long long)ws_all[d];
  const unsigned mlo = __builtin_amdgcn_readfirstlane((unsigned)mp), mhi = __builtin_amdgcn_readfirstlane((unsigned)(mp >> 32));
  const unsigned dlo = __builtin_amdgcn_readfirstlane((unsigned)dp), dhi = __builtin_amdgcn_readfirstlane((unsigned)(dp >> 32));
  asm volatile("s_nop 4\n\t" GEMM_ASM
    :
    : [wlo]"s"((unsigned)wp), [whi]"s"((unsigned)(wp >> 32)), [olo]"s"((unsigned)op), [ohi]"s"((unsigned)(op >> 32)),
      [xilo]"s"((unsigned)xi), [xihi]"s"((unsigned)(xi >> 32)), [mlo]"s"(mlo), [mhi]"s"(mhi), [dlo]"s"(dlo), [dhi]"s"(dhi),
      [rank]"s"(rank), [wave]"s"(wave), [slot]"s"(slot), [dst]"s"(d), [pushc]"s"(pushc), [flagc]"s"(flagc), [dstoff]"s"(dstoff), [flgoff]"s"(flgoff), [dbgoff]"s"(dbgoff), [avinit]"s"(avinit), [ctroff]"s"(ctroff),
      [woff]"v"(woff), [ldsrd]"v"(ldsrd), [ldswr]"v"(ldswr), [xoff]"v"(xoff), [ooff]"v"(ooff), [pfoff]"v"(pfoff),
      [uid4]"v"(uid4), [poff]"v"(poff), [ldsdec]"v"(ldsdec), [ldsd0]"v"(ldsd0), [xioff]"v"(xioff), [pshoff]"v"(pshoff), [ldsstg]"v"(ldsstg)
    : GEMM_CLOBBERS);
}

extern "C" void kernel_launch(int rank, int nranks, ncclComm_t comm,
                              hipStream_t stream,
                              void* const* d_in, const int* in_sizes, int n_in,
                              void* d_out, int out_size,
                              const int* in_dtypes, int out_dtype,
                              void* const* d_out_all,
                              void* const* d_ws_all, size_t ws_bytes) {
  if (nranks != 8 || rank < 0 || rank >= 8 || ws_bytes < (size_t)CTRL_BYTES + 2u * (size_t)XS_BYTES) {
    fprintf(stderr, "unexpected configuration\n");
    return;
  }
  static bool init[8] = {false, false, false, false, false, false, false, false};
  if (!init[rank]) {
    HIP_CHECK(hipFuncSetAttribute((const void*)a2a_gemm_kernel, hipFuncAttributeMaxDynamicSharedMemorySize, (int)LDS_BYTES));
    init[rank] = true;
  }
  const float* x = (const float*)d_in[0];
  const float* W = (const float*)d_in[1];
  a2a_gemm_kernel<<<256, 256, LDS_BYTES, stream>>>(rank, x, W, (float*)d_out, d_ws_all);
  HIP_CHECK(hipGetLastError());
}
```
